```python
import numpy as np

kernel_name = "gpu_gemm_ar_m4096_k4096_n4096_w8a8_dqrelu"


ABSMAX_RTOL = 0.001

COMPUTE_DTYPE = "int8"
EPILOGUE = "dequant_relu"
IN_QUANT = "int8"
OUT_QUANT = None
IN_DTYPES = {"x": "int8", "w_mat": "int8",
             "scale_x": "float32", "scale_w": "float32"}
WORLD_SIZE = 8

def setup_inputs():

    x = np.random.randint(-127, 128, size=(4096, 4096)).astype(np.int8)
    w_mat = np.random.randint(-127, 128, size=(4096, 4096)).astype(np.int8)
    scale_x = (0.02 * (1.0 + np.random.rand(1))).astype(np.float32)
    scale_w = (0.02 * (1.0 + np.random.rand(1))).astype(np.float32)
    return {"x": x, "w_mat": w_mat, "scale_x": scale_x, "scale_w": scale_w}

def reference(x, w_mat, scale_x, scale_w):
    acc = x.astype(np.float64) @ w_mat.astype(np.float64)
    y = (acc.astype(np.float32) * (scale_x * scale_w)).astype(np.float32)
    return np.maximum(y, 0.0).astype(np.float32)
```

```cpp
#include <hip/hip_runtime.h>
#include <rccl/rccl.h>
#include <cstdio>
#include <utility>

#define HIP_CHECK(expr) do { hipError_t _e = (expr); if (_e != hipSuccess) { \
    fprintf(stderr, "HIP error %s at %s:%d\n", hipGetErrorString(_e), __FILE__, __LINE__); \
    return; } } while (0)

namespace {

typedef unsigned u32;
typedef unsigned long long u64;
typedef u32 u32x2 __attribute__((ext_vector_type(2)));
typedef u32 u32x4 __attribute__((ext_vector_type(4)));
typedef int i32x4 __attribute__((ext_vector_type(4)));
typedef int i32x16 __attribute__((ext_vector_type(16)));
typedef float f32x4 __attribute__((ext_vector_type(4)));
#define LAS __attribute__((address_space(3)))
#define GAS __attribute__((address_space(1)))

constexpr int NR = 8;
constexpr int MDIM = 4096, NDIM = 4096, KSH = 512;
constexpr int NWG = 256, NTHR = 768;
constexpr u32 NQ = 64;
constexpr u32 UNIT = 16384;
constexpr u32 BLK_BYTES = NQ * UNIT;
constexpr u32 PAR_BYTES = 32 * BLK_BYTES;
constexpr size_t WS_CTR = 0;
constexpr size_t WS_TMO = 16384;
constexpr size_t WS_FLAG = 32768;
constexpr size_t WS_TOK = 65536;
constexpr size_t WS_DBG = 131072;
constexpr size_t WS_DATA = 1u << 20;
constexpr size_t WS_NEED = WS_DATA + 2ull * PAR_BYTES;
constexpr u64 DEADLINE = 40000000ull;
constexpr u64 TOK_LIMIT = 20000ull;
constexpr u32 SPIN_LIMIT = 10000000u;
constexpr u32 NSLOT = 4;
constexpr u32 SLOT = 2 * UNIT;
constexpr u32 RING = NSLOT * SLOT;
constexpr u32 L_SYNC = RING, L_READY = RING + 128, L_ABORT = RING + 136;
constexpr u32 LDS_BYTES = RING + 256;
#define PUSH_D 10
#define LD_AUX 17

struct Params {
  const char* x;
  const char* w;
  const float* sx;
  const float* sw;
  float* out;
  char* ws[NR];
};
#define DBG(bit) 0
#define DEPTH ((u32)PUSH_D)
#define STAMP(idx) do {} while (0)

template <int N>
__device__ __forceinline__ void wait_vm() { asm volatile("s_waitcnt vmcnt(%c0)" ::"n"(N) : "memory"); }
__device__ __forceinline__ void wait_lgkm0() { asm volatile("s_waitcnt lgkmcnt(0)" ::: "memory"); }
__device__ __forceinline__ u64 rt() { return __builtin_amdgcn_s_memrealtime(); }
__device__ __forceinline__ u32 rfl(u32 v) { return __builtin_amdgcn_readfirstlane(v); }
__device__ __forceinline__ __amdgpu_buffer_rsrc_t mkrsrc(const void* p, u32 bytes) {
  return __builtin_amdgcn_make_buffer_rsrc((void*)p, 0, (int)bytes, 0x00020000);
}
#define PIN() do { asm volatile("" ::: "memory"); __builtin_amdgcn_sched_barrier(0); } while (0)

template <bool ISW>
__device__ __forceinline__ void load_unit(u32x4 (&R)[16], __amdgpu_buffer_rsrc_t rs, u32 lo, u32 s) {
#pragma unroll
  for (int i = 0; i < 16; ++i) {
    if constexpr (ISW) R[i] = __builtin_amdgcn_raw_buffer_load_b128(rs, lo, (s * 64 + i) * NDIM, 0);
    else R[i] = __builtin_amdgcn_raw_buffer_load_b128(rs, lo + (i & 1) * 32, s * 64 + (i >> 1) * 32 * KSH, 0);
  }
}

__device__ __forceinline__ void transpose16(u32x4 (&T)[16], const u32x4 (&R)[16]) {
#pragma unroll
  for (int d = 0; d < 4; ++d)
#pragma unroll
    for (int e = 0; e < 4; ++e) {
      const u32 a0 = R[4 * e][d], a1 = R[4 * e + 1][d], a2 = R[4 * e + 2][d], a3 = R[4 * e + 3][d];
      const u32 l01 = __builtin_amdgcn_perm(a1, a0, 0x05010400u), h01 = __builtin_amdgcn_perm(a1, a0, 0x07030602u);
      const u32 l23 = __builtin_amdgcn_perm(a3, a2, 0x05010400u), h23 = __builtin_amdgcn_perm(a3, a2, 0x07030602u);
      T[4 * d + 0][e] = __builtin_amdgcn_perm(l23, l01, 0x05040100u);
      T[4 * d + 1][e] = __builtin_amdgcn_perm(l23, l01, 0x07060302u);
      T[4 * d + 2][e] = __builtin_amdgcn_perm(h23, h01, 0x05040100u);
      T[4 * d + 3][e] = __builtin_amdgcn_perm(h23, h01, 0x07060302u);
    }
}

template <bool ISW>
__device__ __forceinline__ void pusher(const Params& p, int lane) {
  const u32 b = blockIdx.x, delta = b & 7, pos = b >> 3, blk = pos >> 1, blkid = (ISW ? 16 : 0) + blk;
  char* const ws0 = p.ws[0];
  const __amdgpu_buffer_rsrc_t rs = mkrsrc(ISW ? p.w : p.x, (u32)MDIM * KSH);
  u32 lo;
  if (ISW) lo = (u32)(lane >> 4) * 16 * NDIM + blk * 256 + (lane & 15) * 16;
  else lo = (blk * 256 + (lane & 31)) * KSH + (lane >> 5) * 16;
  u32x4 R[16], T[16];
  GAS u32* ctr = (GAS u32*)(ws0 + WS_CTR) + b * 16 + 10;
  const u32 cv = __hip_atomic_load(ctr, __ATOMIC_RELAXED, __HIP_MEMORY_SCOPE_AGENT);
  load_unit<ISW>(R, rs, lo, 0);
  __builtin_amdgcn_s_barrier();
  const u32 call = rfl(cv) + 1;
  char* dws = p.ws[delta];
  const __amdgpu_buffer_rsrc_t rd = mkrsrc(dws + WS_DATA + (size_t)(call & 1) * PAR_BYTES + (size_t)blkid * BLK_BYTES, BLK_BYTES);
  GAS u32* flags = (GAS u32*)(dws + WS_FLAG) + blkid * NQ + delta;
  GAS u32* tok_mine = (GAS u32*)(ws0 + WS_TOK) + b * 32;
  const u32 depth = DEPTH;
  GAS u32* tok_pred = (GAS u32*)(ws0 + WS_TOK) + ((((pos - depth) & 31) << 3) | delta) * 32;
  const u32 so = lane * 16 + delta * UNIT;
  const bool skip = (DBG(0) && delta != 0) || (DBG(2) && delta == 0);
  for (u32 s = 0; s < 8 && !skip; ++s) {
    if constexpr (ISW) transpose16(T, R);
    else {
#pragma unroll
      for (int i = 0; i < 16; ++i) T[i] = R[i];
    }
#pragma unroll
    for (int i = 0; i < 16; ++i) asm volatile("" : "+v"(T[i]));
    if (s + 1 < 8) load_unit<ISW>(R, rs, lo, s + 1);
    const u32 need = call * 8 + s + (pos >= depth ? 1 : 0);
    if (delta != 0 && need > call * 8) {
      const u64 t0 = rt();
      while (rfl(__hip_atomic_load(tok_pred, __ATOMIC_RELAXED, __HIP_MEMORY_SCOPE_AGENT)) < need) {
        if (rt() - t0 > TOK_LIMIT) break;
        __builtin_amdgcn_s_sleep(3);
      }
    }
#pragma unroll
    for (int i = 0; i < 16; ++i)
      __builtin_amdgcn_raw_buffer_store_b128(T[i], rd, ((ISW ? so ^ ((i >> 2) << 6) : so) + s * 8 * UNIT + (i >> 2) * 4096) + (i & 3) * 1024, 0, 17);
    wait_vm<0>();
    if (lane == 0) {
      __hip_atomic_store(tok_mine, call * 8 + s + 1, __ATOMIC_RELAXED, __HIP_MEMORY_SCOPE_AGENT);
      __hip_atomic_store(flags + s * 8, call, __ATOMIC_RELAXED, __HIP_MEMORY_SCOPE_SYSTEM);
    }
  }
  if (lane == 0) __hip_atomic_store(ctr, call, __ATOMIC_RELAXED, __HIP_MEMORY_SCOPE_AGENT);
  STAMP(1);
}

__global__ void __launch_bounds__(NTHR) fused_ag_gemm(Params p) {
  __shared__ __attribute__((aligned(1024))) char lds[LDS_BYTES];
  LAS char* L = (LAS char*)lds;
  volatile LAS u32* READY = (volatile LAS u32*)(L + L_READY);
  volatile LAS u32* ABORT = (volatile LAS u32*)(L + L_ABORT);
  const int lane = threadIdx.x & 63;
  const int wave = rfl(threadIdx.x >> 6);
  const u32 b = blockIdx.x;
  char* const ws0 = p.ws[0];

  if (wave == 10) {
    __builtin_amdgcn_s_setprio(3);
    if ((b >> 3) & 1) pusher<true>(p, lane);
    else pusher<false>(p, lane);
    return;
  }

  const u32 j = b >> 3;
  const u64 tab = j < 16 ? (j < 8 ? 0x0b030a0908020100ull : 0x0514131211100c04ull) : (j < 24 ? 0x1a1918160e06150dull : 0x1f170f071e1d1c1bull);
  const u32 te = (u32)(tab >> ((j & 7) * 8)) & 31;
  const u32 rb = (te >> 3) * 4 + ((b & 7) >> 1), cb = (te & 7) * 2 + (b & 1);

  if (wave < 8) {
    if (wave == 0) ((volatile LAS u32*)(L + RING))[lane] = 0;
    wait_lgkm0();
    __builtin_amdgcn_s_barrier();
    if (wave == 0) STAMP(0);
    const int wr = wave >> 1, wc = wave & 1;
    u32 aaddr, baddr;
    {
      const int h = lane >> 5, c = lane & 31;
      aaddr = wr * 2048 + lane * 16;
      baddr = UNIT + (c & 3) * 4096 + h * 256 + (((8 * wc + (c >> 2)) ^ ((c & 3) << 2)) * 16);
    }
    const u32 saddr = L_SYNC + wave * 16;
    i32x16 acc[2][4];
#define MM(t, A, B) "v_mfma_i32_32x32x32_i8 v[" #t "*16:" #t "*16+15], v[" #A ":" #A "+3], v[" #B ":" #B "+3], v[" #t "*16:" #t "*16+15]\n" \
                    "v_mfma_i32_32x32x32_i8 v[" #t "*16+64:" #t "*16+79], v[" #A "+4:" #A "+7], v[" #B ":" #B "+3], v[" #t "*16+64:" #t "*16+79]\n"
#define RDA(A, kh) "ds_read_b128 v[" #A ":" #A "+3], v160 offset:" #kh "*1024\n" \
                   "ds_read_b128 v[" #A "+4:" #A "+7], v160 offset:" #kh "*1024+8192\n"
#define RDB(t, kh) "ds_read_b128 v[144+4*" #t ":147+4*" #t "], v161 offset:" #t "*1024+" #kh "*512\n"
#define WL(n) "s_waitcnt lgkmcnt(" #n ")\n"
    u32 sq, sav, ssp;
    asm volatile(
        "v_mov_b32 v128, 0\n v_mov_b32 v129, 0\n v_mov_b32 v130, 0\n v_mov_b32 v131, 0\n"
        "s_mov_b32 %[q], 0\n"
        "s_mov_b32 %[sp], 0\n"
        "s_nop 1\n"
        "v_mfma_i32_32x32x32_i8 v[0:15], v[128:131], v[128:131], 0\n"
        "v_mfma_i32_32x32x32_i8 v[16:31], v[128:131], v[128:131], 0\n"
        "v_mfma_i32_32x32x32_i8 v[32:47], v[128:131], v[128:131], 0\n"
        "v_mfma_i32_32x32x32_i8 v[48:63], v[128:131], v[128:131], 0\n"
        "v_mfma_i32_32x32x32_i8 v[64:79], v[128:131], v[128:131], 0\n"
        "v_mfma_i32_32x32x32_i8 v[80:95], v[128:131], v[128:131], 0\n"
        "v_mfma_i32_32x32x32_i8 v[96:111], v[128:131], v[128:131], 0\n"
        "v_mfma_i32_32x32x32_i8 v[112:127], v[128:131], v[128:131], 0\n"
        "Lspin%=:\n"
        "ds_read_b64 v[164:165], v162\n"
        WL(0)
        "v_min_u32 v164, v164, v165\n"
        "s_nop 0\n"
        "v_readfirstlane_b32 %[av], v164\n"
        "s_cmp_gt_u32 %[av], %[q]\n"
        "s_cbranch_scc1 Lgo%=\n"
        "s_add_u32 %[sp], %[sp], 1\n"
        "s_cmp_gt_u32 %[sp], %[lim]\n"
        "s_cbranch_scc1 Lend%=\n"
        "s_sleep 1\n"
        "s_branch Lspin%=\n"
        "Lgo%=:\n"
        RDA(128, 0) RDB(0, 0) RDB(1, 0) RDB(2, 0)
        "Lstep%=:\n"
        RDB(3, 0) RDA(136, 1)
        "s_add_u32 %[q], %[q], 1\n"
        "v_mov_b32 v163, %[q]\n"
        WL(5) MM(0, 128, 144)
        RDB(0, 1) WL(5) MM(1, 128, 148)
        RDB(1, 1) WL(5) MM(2, 128, 152)
        RDB(2, 1) "ds_read_b64 v[164:165], v162\n" WL(6) MM(3, 128, 156)
        RDB(3, 1) "ds_write_b32 v162, v163 offset:8\n"
        "v_add_u32 v160, 0x8000, v160\n"
        "v_add_u32 v161, 0x8000, v161\n"
        "v_and_b32 v160, 0x1ffff, v160\n"
        "v_and_b32 v161, 0x1ffff, v161\n"
        WL(5) MM(0, 136, 144)
        WL(2)
        "v_min_u32 v164, v164, v165\n"
        "s_nop 0\n"
        "v_readfirstlane_b32 %[av], v164\n"
        "s_cmp_gt_u32 %[av], %[q]\n"
        "s_cbranch_scc0 Lslow%=\n"
        RDA(128, 0) RDB(0, 0) MM(1, 136, 148)
        RDB(1, 0) MM(2, 136, 152)
        RDB(2, 0) WL(6) MM(3, 136, 156)
        "s_branch Lstep%=\n"
        "Lslow%=:\n"
        MM(1, 136, 148) MM(2, 136, 152) WL(0) MM(3, 136, 156)
        "s_cmp_lt_u32 %[q], 64\n"
        "s_cbranch_scc1 Lspin%=\n"
        "Lend%=:\n"
        "s_nop 15\n s_nop 7\n"
        : "={v[0:15]}"(acc[0][0]), "={v[16:31]}"(acc[0][1]), "={v[32:47]}"(acc[0][2]), "={v[48:63]}"(acc[0][3]),
          "={v[64:79]}"(acc[1][0]), "={v[80:95]}"(acc[1][1]), "={v[96:111]}"(acc[1][2]), "={v[112:127]}"(acc[1][3]),
          "+{v160}"(aaddr), "+{v161}"(baddr), [q] "=&s"(sq), [av] "=&s"(sav), [sp] "=&s"(ssp)
        : "{v162}"(saddr), [lim] "s"(SPIN_LIMIT)
        : "memory", "scc", "v128", "v129", "v130", "v131", "v132", "v133", "v134", "v135", "v136", "v137", "v138", "v139", "v140",
          "v141", "v142", "v143", "v144", "v145", "v146", "v147", "v148", "v149", "v150", "v151", "v152", "v153", "v154", "v155",
          "v156", "v157", "v158", "v159", "v163", "v164", "v165");
    if (wave == 0) STAMP(2);
    float scale = p.sx[0] * p.sw[0];
    if (rfl(ABORT[0]) != 0 || sq != NQ) scale = __builtin_nanf("");
    const __amdgpu_buffer_rsrc_t ro = mkrsrc(p.out, (u32)MDIM * NDIM * 4);
    const u32 vo = (rb * 256 + wr * 32 + (lane >> 5) * 4) * (NDIM * 4) + (cb * 256 + wc * 128) * 4 + (lane & 31) * 16;
#pragma unroll
    for (int m = 0; m < 2; ++m)
#pragma unroll
      for (int r = 0; r < 16; ++r) {
        f32x4 v;
#pragma unroll
        for (int t = 0; t < 4; ++t) v[t] = __builtin_elementwise_maximum((float)acc[m][t][r] * scale, 0.f);
        __builtin_amdgcn_raw_buffer_store_b128(__builtin_bit_cast(u32x4, v), ro,
                                               vo + (m * 128 + (r & 3) + 8 * (r >> 2)) * (NDIM * 4), 0, 16);
      }
    return;
  }

  __builtin_amdgcn_s_setprio(2);
  __builtin_amdgcn_s_barrier();
  GAS u32* ctr = (GAS u32*)(ws0 + WS_CTR) + b * 16 + wave;
  const u32 call = rfl(__hip_atomic_load(ctr, __ATOMIC_RELAXED, __HIP_MEMORY_SCOPE_AGENT)) + 1;
  if (lane == 0) __hip_atomic_store(ctr, call, __ATOMIC_RELAXED, __HIP_MEMORY_SCOPE_AGENT);

  if (wave == 11) {
    GAS u32* fa = (GAS u32*)(ws0 + WS_FLAG) + rb * NQ + lane;
    GAS u32* fb = (GAS u32*)(ws0 + WS_FLAG) + (16 + cb) * NQ + lane;
    GAS u32* tmo = (GAS u32*)(ws0 + WS_TMO);
    const u64 t0 = rt();
    if (DBG(1)) { if (lane == 0) { READY[0] = 64; READY[1] = 64; } return; }
    for (u32 sp = 1;; ++sp) {
      const u32 va = __hip_atomic_load(fa, __ATOMIC_RELAXED, __HIP_MEMORY_SCOPE_SYSTEM);
      const u32 vb = __hip_atomic_load(fb, __ATOMIC_RELAXED, __HIP_MEMORY_SCOPE_SYSTEM);
      const u64 ma = ~__builtin_amdgcn_ballot_w64(va >= call), mb = ~__builtin_amdgcn_ballot_w64(vb >= call);
      const u32 ra = ma ? __builtin_ctzll(ma) : 64, rbb = mb ? __builtin_ctzll(mb) : 64;
      if (lane == 0) { READY[0] = ra; READY[1] = rbb; }
      if (ra == 64 && rbb == 64) break;
      if ((sp & 63) == 0) {
        if (rfl(__hip_atomic_load(tmo, __ATOMIC_RELAXED, __HIP_MEMORY_SCOPE_AGENT)) == call || rt() - t0 > DEADLINE) {
          if (lane == 0) {
            __hip_atomic_store(tmo, call, __ATOMIC_RELAXED, __HIP_MEMORY_SCOPE_AGENT);
            ABORT[0] = 1;
            wait_lgkm0();
            READY[0] = 64; READY[1] = 64;
          }
          break;
        }
      }
      __builtin_amdgcn_s_sleep(2);
    }
    STAMP(4);
    return;
  }

  {
    const u32 op = wave - 8;
    const __amdgpu_buffer_rsrc_t rs =
        mkrsrc(ws0 + WS_DATA + (size_t)(call & 1) * PAR_BYTES + (size_t)(op ? 16 + cb : rb) * BLK_BYTES, BLK_BYTES);
    const u32 waddr = lane < 8 ? L_SYNC + lane * 16 + 8 : L_READY + op * 4;
    const u32 wadd = lane < 8 ? NSLOT : 0;
    const u32 paddr = L_SYNC + (lane & 7) * 16 + op * 4;
    const u32 vo = lane * 16;
    wait_vm<0>();
    u32 issued = 0, landed = 0;
    u64 t0 = 0;
    u32 sp = 0;
    while (landed < 2 * NQ) {
      if (issued < NQ && 2 * issued < (landed & ~1u) + 2 * NSLOT) {
        const u32 d = *(const volatile LAS u32*)(L + waddr);
        if (__builtin_amdgcn_ballot_w64(d + wadd > issued) == ~0ull) {
          PIN();
          LAS char* dst = L + (issued & (NSLOT - 1)) * SLOT + op * UNIT;
          if (!DBG(4)) {
#pragma unroll
            for (int g = 0; g < 4; ++g) {
              __builtin_amdgcn_raw_ptr_buffer_load_lds(rs, (LAS void*)(dst + g * 4096), 16, vo, issued * UNIT + g * 4096, 0, LD_AUX);
              __builtin_amdgcn_raw_ptr_buffer_load_lds(rs, (LAS void*)(dst + g * 4096), 16, vo, issued * UNIT + g * 4096, 1024, LD_AUX);
              __builtin_amdgcn_raw_ptr_buffer_load_lds(rs, (LAS void*)(dst + g * 4096), 16, vo, issued * UNIT + g * 4096, 2048, LD_AUX);
              __builtin_amdgcn_raw_ptr_buffer_load_lds(rs, (LAS void*)(dst + g * 4096), 16, vo, issued * UNIT + g * 4096, 3072, LD_AUX);
            }
          }
          PIN();
          ++issued;
          continue;
        }
      }
      if (landed < 2 * issued) {
        switch (2 * issued - landed) {
          case 1: wait_vm<0>(); break;
          case 2: wait_vm<8>(); break;
          case 3: wait_vm<16>(); break;
          case 4: wait_vm<24>(); break;
          case 5: wait_vm<32>(); break;
          case 6: wait_vm<40>(); break;
          case 7: wait_vm<48>(); break;
          default: wait_vm<56>(); break;
        }
        ++landed;
        if ((landed & 1) == 0) *(volatile LAS u32*)(L + paddr) = landed >> 1;
        continue;
      }
      if ((++sp & 4095) == 0) {
        const u64 t = rt();
        if (t0 == 0) t0 = t;
        if (t - t0 > 2 * DEADLINE) { ABORT[0] = 1; wait_lgkm0(); *(volatile LAS u32*)(L + paddr) = NQ; break; }
      }
      __builtin_amdgcn_s_sleep(1);
    }
    STAMP(5 + op);
  }
}

}

extern "C" void kernel_launch(int rank, int nranks, ncclComm_t comm,
                              hipStream_t stream,
                              void* const* d_in, const int* in_sizes, int n_in,
                              void* d_out, int out_size,
                              const int* in_dtypes, int out_dtype,
                              void* const* d_out_all,
                              void* const* d_ws_all, size_t ws_bytes) {
  if (nranks != NR || n_in != 4 || (unsigned)rank >= (unsigned)NR || ws_bytes < WS_NEED ||
      in_sizes[0] != MDIM * KSH || in_sizes[1] != KSH * NDIM || out_size != MDIM * NDIM) {
    fprintf(stderr, "kernel_launch: unexpected problem shape\n");
    return;
  }
  Params p;
  p.x = (const char*)d_in[0];
  p.w = (const char*)d_in[1];
  p.sx = (const float*)d_in[2];
  p.sw = (const float*)d_in[3];
  p.out = (float*)d_out;
  for (int d = 0; d < NR; ++d) p.ws[d] = (char*)d_ws_all[(rank + d) % NR];
  fused_ag_gemm<<<NWG, NTHR, 0, stream>>>(p);
  HIP_CHECK(hipGetLastError());
}
```

```cpp
#include <hip/hip_runtime.h>
#include <rccl/rccl.h>
#include <cstdio>
#include <utility>

#define HIP_CHECK(expr) do { hipError_t _e = (expr); if (_e != hipSuccess) { \
    fprintf(stderr, "HIP error %s at %s:%d\n", hipGetErrorString(_e), __FILE__, __LINE__); \
    return; } } while (0)

namespace {

typedef unsigned u32;
typedef unsigned long long u64;
typedef u32 u32x2 __attribute__((ext_vector_type(2)));
typedef u32 u32x4 __attribute__((ext_vector_type(4)));
typedef int i32x4 __attribute__((ext_vector_type(4)));
typedef int i32x16 __attribute__((ext_vector_type(16)));
typedef float f32x4 __attribute__((ext_vector_type(4)));
#define LAS __attribute__((address_space(3)))
#define GAS __attribute__((address_space(1)))

constexpr int NR = 8;
constexpr int MDIM = 4096, NDIM = 4096, KSH = 512;
constexpr int NWG = 256, NTHR = 768;
constexpr u32 NQ = 64;
constexpr u32 UNIT = 16384;
constexpr u32 BLK_BYTES = NQ * UNIT;
constexpr u32 PAR_BYTES = 32 * BLK_BYTES;
constexpr size_t WS_CTR = 0;
constexpr size_t WS_TMO = 16384;
constexpr size_t WS_CLAIM = 20480;
constexpr size_t WS_FLAG = 32768;
constexpr size_t WS_DBG = 131072;
constexpr size_t WS_DATA = 1u << 20;
constexpr size_t WS_NEED = WS_DATA + 2ull * PAR_BYTES;
constexpr u64 DEADLINE = 40000000ull;
constexpr u32 SPIN_LIMIT = 10000000u;
constexpr u32 NPUSH = 48, PPL = NPUSH / NR;
constexpr u32 NCONS = 10;
constexpr u32 NSLOT = 4;
constexpr u32 XUNIT = 5 * 4096;
constexpr u32 L_X = NSLOT * UNIT, RING = L_X + NSLOT * XUNIT;
constexpr u32 L_SYNC = RING, L_ABORT = RING + 192, L_TOK = RING + 196, L_DST = RING + 200, L_ORD = RING + 224;
constexpr u32 LDS_BYTES = RING + 256;
#define PUSH_D 8

struct Params {
  const char* src[2];
  const float* sx;
  const float* sw;
  float* out;
  char* ws0;
  char* dst[NR];
  unsigned perm;
};
#define DBG(bit) 0
#define DEPTH ((u32)PUSH_D)
#define ROT 0
#define PFD 2
#define STAMP(idx) do {} while (0)

template <int N>
__device__ __forceinline__ void wait_vm() { asm volatile("s_waitcnt vmcnt(%c0)" ::"n"(N) : "memory"); }
__device__ __forceinline__ void wait_lgkm0() { asm volatile("s_waitcnt lgkmcnt(0)" ::: "memory"); }
__device__ __forceinline__ u64 rt() { return __builtin_amdgcn_s_memrealtime(); }
__device__ __forceinline__ int lane_id() {
  int ln;
  asm volatile("v_mbcnt_lo_u32_b32 %0, -1, 0\n v_mbcnt_hi_u32_b32 %0, -1, %0" : "=v"(ln));
  __builtin_assume(ln >= 0 && ln < 64);
  return ln;
}
__device__ __forceinline__ u32 rfl(u32 v) { return __builtin_amdgcn_readfirstlane(v); }
__device__ __forceinline__ __amdgpu_buffer_rsrc_t mkrsrc(const void* p, u32 bytes) {
  return __builtin_amdgcn_make_buffer_rsrc((void*)p, 0, (int)bytes, 0x00020000);
}
#define PIN() do { asm volatile("" ::: "memory"); __builtin_amdgcn_sched_barrier(0); } while (0)

template <bool ISW>
__device__ __forceinline__ void load_unit(u32x4 (&R)[16], __amdgpu_buffer_rsrc_t rs, u32 lo, u32 s) {
#pragma unroll
  for (int i = 0; i < 16; ++i) {
    if constexpr (ISW) R[i] = __builtin_amdgcn_raw_buffer_load_b128(rs, lo, (s * 64 + i) * NDIM, 0);
    else R[i] = __builtin_amdgcn_raw_buffer_load_b128(rs, lo + (i & 1) * 32, s * 64 + (i >> 1) * 32 * KSH, 0);
  }
}

__device__ __forceinline__ void transpose16(u32x4 (&T)[16], const u32x4 (&R)[16]) {
#pragma unroll
  for (int d = 0; d < 4; ++d)
#pragma unroll
    for (int e = 0; e < 4; ++e) {
      const u32 a0 = R[4 * e][d], a1 = R[4 * e + 1][d], a2 = R[4 * e + 2][d], a3 = R[4 * e + 3][d];
      const u32 l01 = __builtin_amdgcn_perm(a1, a0, 0x05010400u), h01 = __builtin_amdgcn_perm(a1, a0, 0x07030602u);
      const u32 l23 = __builtin_amdgcn_perm(a3, a2, 0x05010400u), h23 = __builtin_amdgcn_perm(a3, a2, 0x07030602u);
      T[4 * d + 0][e] = __builtin_amdgcn_perm(l23, l01, 0x05040100u);
      T[4 * d + 1][e] = __builtin_amdgcn_perm(l23, l01, 0x07060302u);
      T[4 * d + 2][e] = __builtin_amdgcn_perm(h23, h01, 0x05040100u);
      T[4 * d + 3][e] = __builtin_amdgcn_perm(h23, h01, 0x07060302u);
    }
}

__device__ __forceinline__ bool lds_wait_ge(volatile LAS u32* w, int need) {
  for (u32 sp = 0; (int)rfl(*w) < need; ++sp) {
    if (sp > SPIN_LIMIT) return false;
    __builtin_amdgcn_s_sleep(1);
  }
  return true;
}

__device__ __forceinline__ u32 claim_add(__amdgpu_buffer_rsrc_t rc, int lane, u32 slot, int v) {
  return __builtin_amdgcn_raw_ptr_buffer_atomic_add_i32(v, rc, lane * 512 + slot * 64, 0, 0);
}

template <bool ISW>
__device__ __forceinline__ void pusher(const Params& p, LAS char* L, int lane, u32 wave, u32 pref, const char* src,
                                       char* ws0, char* dws, u32 perm) {
  const u32 b = blockIdx.x, j = (b >> 3) ^ (wave & 1);
  volatile LAS u32* TOK = (volatile LAS u32*)(L + L_TOK);
  volatile LAS u32* DST = (volatile LAS u32*)(L + L_DST);
  const __amdgpu_buffer_rsrc_t rs = mkrsrc(src, (u32)MDIM * KSH);
  u32 lo;
  if (ISW) lo = (u32)(lane >> 4) * 16 * NDIM + (lane & 15) * 16;
  else lo = (lane & 31) * KSH + (lane >> 5) * 16;
  constexpr u32 BSTRIDE = ISW ? 256 : 256 * KSH;
  const int PF = PFD;
  u32x4 R[16], T[16];
  GAS u32* ctr = (GAS u32*)(ws0 + WS_CTR) + b * 16 + wave;
  const __amdgpu_buffer_rsrc_t rc = mkrsrc(ws0 + WS_CLAIM + (b >> 3) * 512, 512);
  u32 old = 0;
  if (wave == 0) old = claim_add(rc, lane, pref, 1);
  u32 cv = __hip_atomic_load(ctr, __ATOMIC_RELAXED, __HIP_MEMORY_SCOPE_AGENT);
  PIN();
  if (wave == 0) load_unit<ISW>(R, rs, lo + (j >> 1) * BSTRIDE, 0);
  if (wave == 11) ((volatile LAS u32*)(L + RING))[lane] = 0;
  wait_lgkm0();
  __builtin_amdgcn_s_barrier();
  if (wave != 0 && (int)wave <= PF) load_unit<ISW>(R, rs, lo + ((wave * PPL + j) >> 1) * BSTRIDE, 0);
  PIN();
  asm volatile("" : "+v"(cv), "+v"(old));
  const u32 call = rfl(cv) + 1;
  u32 slot = pref;
  if (wave == 0) {
#pragma nounroll
    for (int k = 0; k < 7 && rfl(old) + 1 != call; ++k) {
      claim_add(rc, lane, slot, -1);
      slot = (slot + 1) & 7;
      old = claim_add(rc, lane, slot, 1);
    }
    *DST = slot + 1;
  } else {
    lds_wait_ge(DST, 1);
    slot = (rfl(*DST) - 1) & 7;
  }
  if (slot != pref) dws = p.dst[slot];
  const u32 delta = (perm >> (slot * 4)) & 7;
  if (wave == 0) STAMP(0);
  const __amdgpu_buffer_rsrc_t rd = mkrsrc(dws + WS_DATA + (size_t)(call & 1) * PAR_BYTES + (ISW ? 16 * BLK_BYTES : 0), 16 * BLK_BYTES);
  GAS u32* flags = (GAS u32*)(dws + WS_FLAG) + (ISW ? 16 * NQ : 0) + delta;
  const u32 depth = delta ? DEPTH : 16;
  const bool skip = (DBG(0) && delta != 0) || (DBG(2) && delta == 0);
  for (u32 m = wave; m * PPL + j < 256 && !skip; m += 12) {
    const u32 n = m * PPL + j, s = n >> 5, blk = (n & 31) >> 1;
    if ((int)m > PF) {
      if (!lds_wait_ge(TOK, (int)m - PF)) break;
      load_unit<ISW>(R, rs, lo + blk * BSTRIDE, s);
    }
    if constexpr (ISW) transpose16(T, R);
    else {
#pragma unroll
      for (int i = 0; i < 16; ++i) T[i] = R[i];
    }
#pragma unroll
    for (int i = 0; i < 16; ++i) asm volatile("" : "+v"(T[i]));
    const u32 so = lane * 16 + (s * 8 + delta) * UNIT + blk * BLK_BYTES;
    wait_vm<0>();
    if (!lds_wait_ge(TOK, (int)m)) break;
#pragma unroll
    for (int i = 0; i < 16; ++i)
      __builtin_amdgcn_raw_buffer_store_b128(T[i], rd, ((ISW ? so ^ ((i >> 2) << 6) : so) + (i >> 2) * 4096) + (i & 3) * 1024, 0, 17);
    if (depth != 16) wait_vm<PUSH_D>();
    *TOK = m + 1;
    wait_vm<0>();
    if (lane == 0) __hip_atomic_store(flags + blk * NQ + s * 8, call, __ATOMIC_RELAXED, __HIP_MEMORY_SCOPE_SYSTEM);
  }
  if (lane == 0) __hip_atomic_store(ctr, call, __ATOMIC_RELAXED, __HIP_MEMORY_SCOPE_AGENT);
}

template <bool ISW>
__device__ __forceinline__ void loader(const Params& p, LAS char* L, int lane, u32 wave, u32 rg, u32 cb) {
  char* const ws0 = p.ws0;
  const u32 b = blockIdx.x;
  GAS u32* ctr = (GAS u32*)(ws0 + WS_CTR) + b * 16 + wave;
  const u32 cv = __hip_atomic_load(ctr, __ATOMIC_RELAXED, __HIP_MEMORY_SCOPE_AGENT);
  __builtin_amdgcn_s_barrier();
  const u32 call = rfl(cv) + 1;
  const __amdgpu_buffer_rsrc_t rs = mkrsrc(ws0 + WS_DATA + (size_t)(call & 1) * PAR_BYTES + (ISW ? 16 * BLK_BYTES : 0), 16 * BLK_BYTES);
  GAS u32* tmo = (GAS u32*)(ws0 + WS_TMO);
  const u32 cl = min((u32)lane, NCONS - 1);
  const u32 waddr = L_SYNC + cl * 16 + 8, paddr = L_SYNC + cl * 16 + (ISW ? 4 : 0);
  const u32 vo = lane * 16;
  u32 issued = 0, landed = 0, sp = 0;
#define LD4 "buffer_load_dwordx4 %[vo], %[rs], %[t] offen sc0 sc1 lds\n"             \
            "buffer_load_dwordx4 %[vo], %[rs], %[t] offen offset:1024 sc0 sc1 lds\n" \
            "buffer_load_dwordx4 %[vo], %[rs], %[t] offen offset:2048 sc0 sc1 lds\n" \
            "buffer_load_dwordx4 %[vo], %[rs], %[t] offen offset:3072 sc0 sc1 lds\n"
#define TARGET NQ
  if constexpr (ISW) {
    const __amdgpu_buffer_rsrc_t rf = mkrsrc(ws0 + WS_FLAG, 32 * NQ * 4);
    const u32 fo = lane * 4;
    const u32 r0 = (rg * 5 >> 2) * NQ * 4, r1 = min((rg * 5 + 4) >> 2, 15u) * NQ * 4, r2 = (16 + cb) * NQ * 4;
    u64 pend = ~0ull, rdy = DBG(1) ? ~0ull : 0, t0 = 0;
    u32 polling = 0, code, f0 = 0, f1 = 0, f2 = 0;
#define NXT "s_add_u32 m0, m0, 0x1000\n s_add_u32 %[t], %[t], 0x1000\n"
#define WLAND(k) "s_waitcnt vmcnt(" #k "+3)\n s_cbranch_scc1 Lpub%=\n s_waitcnt vmcnt(" #k ")\n"
    for (;;) {
      const u32 target = TARGET;
      u32 st, su, vd, ve;
      u64 av;
      asm volatile(
          "s_mov_b32 %[code], 64\n"
          "Ltop%=:\n"
          "s_and_b64 %[av], %[rdy], %[pend]\n"
          "s_cbranch_scc0 Lneed%=\n"
          "Lgot%=:\n"
          "s_add_u32 %[t], %[la], 3\n"
          "s_cmp_lt_u32 %[is], %[t]\n"
          "s_cbranch_scc0 Lland%=\n"
          "ds_read_b32 %[vd], %[wa]\n"
          "s_ff1_i32_b64 %[u], %[av]\n"
          "s_lshl_b32 %[t], %[is], 8\n"
          "s_add_u32 %[t], %[t], %[u]\n"
          "s_add_u32 %[t], %[t], 0x100\n"
          "v_mov_b32 %[ve], %[t]\n"
          "s_and_b32 %[t], %[is], 3\n"
          "s_lshl_b32 m0, %[t], 14\n"
          "s_lshl_b32 %[t], %[t], 2\n"
          "s_waitcnt lgkmcnt(0)\n"
          "v_add_u32 %[vd], 4, %[vd]\n"
          "v_cmp_ge_u32 vcc, %[is], %[vd]\n"
          "s_cbranch_vccnz Lnoslot%=\n"
          "v_mov_b32 %[vd], %[t]\n"
          "v_add_u32 %[vd], %[ord], %[vd]\n"
          "ds_write_b32 %[vd], %[ve]\n"
          "s_bitset0_b64 %[pend], %[u]\n"
          "s_lshl_b32 %[u], %[u], 14\n"
          "s_add_u32 %[t], %[u], %[g0]\n"
          LD4 NXT LD4 NXT LD4 NXT LD4
          "s_add_u32 %[is], %[is], 1\n"
          "s_branch Ltop%=\n"
          "Lneed%=:\n"
          "s_cmp_eq_u64 %[pend], 0\n"
          "s_cbranch_scc1 Lland%=\n"
          "s_cmp_lg_u32 %[po], 0\n"
          "s_cbranch_scc1 Lpolled%=\n"
          "Lpoll%=:\n"
          "buffer_load_dword %[f0], %[fo], %[rf], %[r0] offen sc0 sc1\n"
          "buffer_load_dword %[f1], %[fo], %[rf], %[r1] offen sc0 sc1\n"
          "buffer_load_dword %[f2], %[fo], %[rf], %[r2] offen sc0 sc1\n"
          "s_mov_b32 %[po], 1\n"
          "Lpolled%=:\n"
          "s_cmp_lt_u32 %[la], %[is]\n"
          "s_cbranch_scc1 Lland%=\n"
          "s_waitcnt vmcnt(0)\n"
          "s_mov_b32 %[po], 0\n"
          "v_min3_u32 %[f0], %[f0], %[f1], %[f2]\n"
          "v_cmp_le_u32 %[rdy], %[call], %[f0]\n"
          "s_nop 0\n"
          "s_and_b64 %[av], %[rdy], %[pend]\n"
          "s_cbranch_scc1 Lgot%=\n"
          "s_sub_u32 %[code], %[code], 1\n"
          "s_cmp_eq_u32 %[code], 0\n"
          "s_cbranch_scc1 Lend%=\n"
          "s_sleep 2\n"
          "s_branch Lpoll%=\n"
          "Lnoslot%=:\n"
          "s_cmp_lt_u32 %[la], %[is]\n"
          "s_cbranch_scc1 Lland%=\n"
          "s_add_u32 %[sp], %[sp], 1\n"
          "s_cmp_gt_u32 %[sp], %[lim]\n"
          "s_cbranch_scc1 Lend%=\n"
          "s_sleep 1\n"
          "s_branch Ltop%=\n"
          "Lland%=:\n"
          "s_sub_u32 %[t], %[is], %[la]\n"
          "s_add_u32 %[la], %[la], 1\n"
          "v_mov_b32 %[vd], %[la]\n"
          "s_cmp_eq_u32 %[t], 1\n"
          "s_cbranch_scc1 Lw1%=\n"
          "s_cmp_eq_u32 %[t], 2\n"
          "s_cbranch_scc1 Lw2%=\n"
          "s_cmp_lg_u32 %[po], 0\n"
          WLAND(32)
          "s_branch Lpub%=\n"
          "Lw2%=:\n"
          "s_cmp_lg_u32 %[po], 0\n"
          WLAND(16)
          "s_branch Lpub%=\n"
          "Lw1%=:\n"
          "s_cmp_lg_u32 %[po], 0\n"
          WLAND(0)
          "Lpub%=:\n"
          "ds_write_b32 %[pa], %[vd]\n"
          "s_cmp_lt_u32 %[la], %[tg]\n"
          "s_cbranch_scc1 Ltop%=\n"
          "s_mov_b32 %[code], 200\n"
          "Lend%=:\n"
          : [rdy] "+s"(rdy), [pend] "+s"(pend), [is] "+s"(issued), [la] "+s"(landed), [po] "+s"(polling), [sp] "+s"(sp),
            [code] "=&s"(code), [t] "=&s"(st), [u] "=&s"(su), [av] "=&s"(av), [vd] "=&v"(vd), [ve] "=&v"(ve),
            [f0] "+{v100}"(f0), [f1] "+{v101}"(f1), [f2] "+{v102}"(f2)
          : [vo] "v"(vo), [wa] "v"(waddr), [pa] "v"(paddr), [fo] "v"(fo), [rs] "s"(rs), [rf] "s"(rf), [g0] "s"(cb * BLK_BYTES),
            [r0] "s"(r0), [r1] "s"(r1), [r2] "s"(r2), [call] "s"(call), [lim] "s"(SPIN_LIMIT), [tg] "s"(target), [ord] "n"(L_ORD)
          : "memory", "scc", "vcc");
      if (landed >= NQ || sp > SPIN_LIMIT) break;
      const u64 t = rt();
      if (t0 == 0) t0 = t;
      const u32 tv = rfl(__hip_atomic_load(tmo, __ATOMIC_RELAXED, __HIP_MEMORY_SCOPE_AGENT));
      if (tv == call || t - t0 > DEADLINE) break;
    }
#undef NXT
  } else {
    u32 goff[5];
#pragma unroll
    for (int g = 0; g < 5; ++g) {
      const u32 strip = min(rg * 5 + g, 63u);
      goff[g] = (strip >> 2) * BLK_BYTES + (strip & 3) * 4096;
    }
#define NXT(g) "s_add_u32 m0, m0, 0x1000\n s_add_u32 %[t], %[u], %[" #g "]\n"
    u32 st, su, vd;
    asm volatile(
        "Ltop%=:\n"
        "s_add_u32 %[t], %[la], 3\n"
        "s_min_u32 %[t], %[t], 64\n"
        "s_cmp_lt_u32 %[is], %[t]\n"
        "s_cbranch_scc0 Lland%=\n"
        "s_and_b32 %[t], %[is], 3\n"
        "s_lshl_b32 %[u], %[t], 2\n"
        "v_mov_b32 %[vd], %[u]\n"
        "v_add_u32 %[vd], %[ord], %[vd]\n"
        "ds_read_b32 %[vd], %[vd]\n"
        "s_mul_i32 %[t], %[t], %[stride]\n"
        "s_add_u32 m0, %[t], %[base]\n"
        "s_waitcnt lgkmcnt(0)\n"
        "v_readfirstlane_b32 %[u], %[vd]\n"
        "s_lshr_b32 %[t], %[u], 8\n"
        "s_cmp_gt_u32 %[t], %[is]\n"
        "s_cbranch_scc0 Lno%=\n"
        "s_cmp_gt_u32 %[t], 64\n"
        "s_cbranch_scc1 Lend%=\n"
        "s_and_b32 %[u], %[u], 63\n"
        "s_lshl_b32 %[u], %[u], 14\n"
        "s_add_u32 %[t], %[u], %[g0]\n"
        LD4 NXT(g1) LD4 NXT(g2) LD4 NXT(g3) LD4 NXT(g4) LD4
        "s_add_u32 %[is], %[is], 1\n"
        "s_branch Ltop%=\n"
        "Lno%=:\n"
        "s_cmp_lt_u32 %[la], %[is]\n"
        "s_cbranch_scc1 Lland%=\n"
        "s_add_u32 %[sp], %[sp], 1\n"
        "s_cmp_gt_u32 %[sp], %[lim]\n"
        "s_cbranch_scc1 Lend%=\n"
        "s_sleep 1\n"
        "s_branch Ltop%=\n"
        "Lland%=:\n"
        "s_sub_u32 %[t], %[is], %[la]\n"
        "s_add_u32 %[la], %[la], 1\n"
        "v_mov_b32 %[vd], %[la]\n"
        "s_waitcnt vmcnt(40)\n"
        "s_cmp_eq_u32 %[t], 3\n"
        "s_cbranch_scc1 Lpub%=\n"
        "s_waitcnt vmcnt(20)\n"
        "s_cmp_eq_u32 %[t], 2\n"
        "s_cbranch_scc1 Lpub%=\n"
        "s_waitcnt vmcnt(0)\n"
        "Lpub%=:\n"
        "ds_write_b32 %[pa], %[vd]\n"
        "s_cmp_lt_u32 %[la], 64\n"
        "s_cbranch_scc1 Ltop%=\n"
        "Lend%=:\n"
        : [is] "+s"(issued), [la] "+s"(landed), [sp] "+s"(sp), [t] "=&s"(st), [u] "=&s"(su), [vd] "=&v"(vd)
        : [vo] "v"(vo), [pa] "v"(paddr), [rs] "s"(rs), [g0] "s"(goff[0]), [g1] "s"(goff[1]), [g2] "s"(goff[2]),
          [g3] "s"(goff[3]), [g4] "s"(goff[4]), [lim] "s"(SPIN_LIMIT), [stride] "n"(XUNIT), [base] "n"(L_X), [ord] "n"(L_ORD)
        : "memory", "scc");
#undef NXT
  }
  if (landed < NQ) {
    *(volatile LAS u32*)(L + L_ABORT) = 1;
    *(volatile LAS u32*)(L + L_ORD + (lane & 3) * 4) = ~0u;
    wait_lgkm0();
    *(volatile LAS u32*)(L + L_SYNC + cl * 16) = NQ;
    *(volatile LAS u32*)(L + L_SYNC + cl * 16 + 4) = NQ;
    if (lane == 0) __hip_atomic_store(tmo, call, __ATOMIC_RELAXED, __HIP_MEMORY_SCOPE_AGENT);
  }
  if (lane == 0) __hip_atomic_store(ctr, call, __ATOMIC_RELAXED, __HIP_MEMORY_SCOPE_AGENT);
  STAMP(ISW ? 6 : 5);
}

__global__ void __launch_bounds__(NTHR) fused_ag_gemm(Params p) {
  __shared__ __attribute__((aligned(1024))) char lds[LDS_BYTES];
  LAS char* L = (LAS char*)lds;
  volatile LAS u32* ABORT = (volatile LAS u32*)(L + L_ABORT);
  const int wave = rfl(threadIdx.x >> 6);
  const u32 b = blockIdx.x;
  char* const ws0 = p.ws0;

  if (b < NPUSH) {
    const u32 isw = ((b >> 3) ^ wave) & 1;
    const u32 pref = (__builtin_amdgcn_s_getreg((3 << 11) | 20) + ROT) & 7;
    const char* src = p.src[isw];
    char *own = p.ws0, *dws = p.dst[pref];
    u32 perm = p.perm;
    asm volatile("" : "+s"(src), "+s"(own), "+s"(dws), "+s"(perm));
    if (isw) pusher<true>(p, L, lane_id(), wave, pref, src, own, dws, perm);
    else pusher<false>(p, L, lane_id(), wave, pref, src, own, dws, perm);
    return;
  }

  const u32 ti = (b >> 3) - PPL;
  const u32 rg = ti < 24 ? (ti >> 2) * 2 + (b & 1) : 12;
  const u32 cb = (ti < 24 ? ti & 3 : (b & 1) * 2 + ti - 24) * 4 + ((b & 7) >> 1);

  if (wave >= (int)NCONS) {
    __builtin_amdgcn_s_setprio(2);
    if (wave == (int)NCONS) loader<false>(p, L, lane_id(), wave, rg, cb);
    else loader<true>(p, L, lane_id(), wave, rg, cb);
    return;
  }

  {
    int lane = lane_id();
    if (wave == 0) ((volatile LAS u32*)(L + RING))[lane] = 0;
    wait_lgkm0();
    __builtin_amdgcn_s_barrier();
    if (wave == 0) STAMP(0);
    const int wr = wave >> 1, wc = wave & 1;
    u32 aaddr, baddr;
    {
      const int h = lane >> 5, c = lane & 31;
      aaddr = L_X + wr * 4096 + lane * 16;
      baddr = (c & 3) * 4096 + h * 256 + (((8 * wc + (c >> 2)) ^ ((c & 3) << 2)) * 16);
    }
    const u32 saddr = L_SYNC + wave * 16;
    const __amdgpu_buffer_rsrc_t ro = mkrsrc(p.out, (u32)MDIM * NDIM * 4);
    const u32 so = (rg * 320 + wr * 64) * (NDIM * 4) + (cb * 256 + wc * 128) * 4;
    u32 touch = __builtin_amdgcn_raw_buffer_load_b32(ro, so + lane * (NDIM * 4), 0, 0);
    float scale = __builtin_bit_cast(float, rfl(__builtin_bit_cast(u32, p.sx[0] * p.sw[0])));
    asm volatile("" : "+s"(scale) : "v"(touch));
    i32x16 acc[2][4];
#define MM(t, A, B) "v_mfma_i32_32x32x32_i8 v[" #t "*16:" #t "*16+15], v[" #A ":" #A "+3], v[" #B ":" #B "+3], v[" #t "*16:" #t "*16+15]\n" \
                    "v_mfma_i32_32x32x32_i8 v[" #t "*16+64:" #t "*16+79], v[" #A "+4:" #A "+7], v[" #B ":" #B "+3], v[" #t "*16+64:" #t "*16+79]\n"
#define RDA(A, kh) "ds_read_b128 v[" #A ":" #A "+3], v160 offset:" #kh "*1024\n" \
                   "ds_read_b128 v[" #A "+4:" #A "+7], v160 offset:" #kh "*1024+2048\n"
#define RDB(t, kh) "ds_read_b128 v[144+4*" #t ":147+4*" #t "], v161 offset:" #t "*1024+" #kh "*512\n"
#define WL(n) "s_waitcnt lgkmcnt(" #n ")\n"
    u32 sq, sav, ssp;
    asm volatile(
        "v_mov_b32 v128, 0\n v_mov_b32 v129, 0\n v_mov_b32 v130, 0\n v_mov_b32 v131, 0\n"
        "s_mov_b32 %[q], 0\n"
        "s_mov_b32 %[sp], 0\n"
        "s_nop 1\n"
        "v_mfma_i32_32x32x32_i8 v[0:15], v[128:131], v[128:131], 0\n"
        "v_mfma_i32_32x32x32_i8 v[16:31], v[128:131], v[128:131], 0\n"
        "v_mfma_i32_32x32x32_i8 v[32:47], v[128:131], v[128:131], 0\n"
        "v_mfma_i32_32x32x32_i8 v[48:63], v[128:131], v[128:131], 0\n"
        "v_mfma_i32_32x32x32_i8 v[64:79], v[128:131], v[128:131], 0\n"
        "v_mfma_i32_32x32x32_i8 v[80:95], v[128:131], v[128:131], 0\n"
        "v_mfma_i32_32x32x32_i8 v[96:111], v[128:131], v[128:131], 0\n"
        "v_mfma_i32_32x32x32_i8 v[112:127], v[128:131], v[128:131], 0\n"
        "Lspin%=:\n"
        "ds_read_b64 v[164:165], v162\n"
        WL(0)
        "v_min_u32 v164, v164, v165\n"
        "s_nop 0\n"
        "v_readfirstlane_b32 %[av], v164\n"
        "s_cmp_gt_u32 %[av], %[q]\n"
        "s_cbranch_scc1 Lgo%=\n"
        "s_add_u32 %[sp], %[sp], 1\n"
        "s_cmp_gt_u32 %[sp], %[lim]\n"
        "s_cbranch_scc1 Lend%=\n"
        "s_sleep 1\n"
        "s_branch Lspin%=\n"
        "Lgo%=:\n"
        RDA(128, 0) RDB(0, 0) RDB(1, 0) RDB(2, 0)
        "Lstep%=:\n"
        RDB(3, 0) RDA(136, 1)
        "s_add_u32 %[q], %[q], 1\n"
        "v_mov_b32 v163, %[q]\n"
        WL(5) MM(0, 128, 144)
        RDB(0, 1) WL(5) MM(1, 128, 148)
        RDB(1, 1) WL(5) MM(2, 128, 152)
        RDB(2, 1) "ds_read_b64 v[164:165], v162\n" WL(6) MM(3, 128, 156)
        RDB(3, 1) "ds_write_b32 v162, v163 offset:8\n"
        "s_and_b32 %[av], %[q], 3\n"
        "s_cselect_b32 %[av], 0x5000, %[wrap]\n"
        "v_add_u32 v161, 0x4000, v161\n"
        "v_add_u32 v160, %[av], v160\n"
        "v_and_b32 v161, 0xffff, v161\n"
        WL(5) MM(0, 136, 144)
        WL(2)
        "v_min_u32 v164, v164, v165\n"
        "s_nop 0\n"
        "v_readfirstlane_b32 %[av], v164\n"
        "s_cmp_gt_u32 %[av], %[q]\n"
        "s_cbranch_scc0 Lslow%=\n"
        RDA(128, 0) RDB(0, 0) MM(1, 136, 148)
        RDB(1, 0) MM(2, 136, 152)
        RDB(2, 0) WL(6) MM(3, 136, 156)
        "s_branch Lstep%=\n"
        "Lslow%=:\n"
        MM(1, 136, 148) MM(2, 136, 152) WL(0) MM(3, 136, 156)
        "s_cmp_lt_u32 %[q], 64\n"
        "s_cbranch_scc1 Lspin%=\n"
        "Lend%=:\n"
        "s_nop 15\n s_nop 7\n"
        : "={v[0:15]}"(acc[0][0]), "={v[16:31]}"(acc[0][1]), "={v[32:47]}"(acc[0][2]), "={v[48:63]}"(acc[0][3]),
          "={v[64:79]}"(acc[1][0]), "={v[80:95]}"(acc[1][1]), "={v[96:111]}"(acc[1][2]), "={v[112:127]}"(acc[1][3]),
          "+{v160}"(aaddr), "+{v161}"(baddr), [q] "=&s"(sq), [av] "=&s"(sav), [sp] "=&s"(ssp)
        : "{v162}"(saddr), [lim] "s"(SPIN_LIMIT), [wrap] "s"(0u - 3 * XUNIT)
        : "memory", "scc", "v128", "v129", "v130", "v131", "v132", "v133", "v134", "v135", "v136", "v137", "v138", "v139", "v140",
          "v141", "v142", "v143", "v144", "v145", "v146", "v147", "v148", "v149", "v150", "v151", "v152", "v153", "v154", "v155",
          "v156", "v157", "v158", "v159", "v163", "v164", "v165");
    lane = lane_id();
    if (wave == 0) STAMP(2);
    if (rfl(ABORT[0]) != 0 || sq != NQ) scale = __builtin_nanf("");
    if (rg * 5 + wr >= 64) return;
    const u32 vo = so + (lane >> 5) * 4 * (NDIM * 4) + (lane & 31) * 16;
#pragma unroll
    for (int m = 0; m < 2; ++m)
#pragma unroll
      for (int r = 0; r < 16; ++r) {
        f32x4 v;
#pragma unroll
        for (int t = 0; t < 4; ++t) v[t] = __builtin_elementwise_maximum((float)acc[m][t][r] * scale, 0.f);
        __builtin_amdgcn_raw_buffer_store_b128(__builtin_bit_cast(u32x4, v), ro,
                                               vo + (m * 32 + (r & 3) + 8 * (r >> 2)) * (NDIM * 4), 0, 16);
      }
  }
}

}

extern "C" void kernel_launch(int rank, int nranks, ncclComm_t comm,
                              hipStream_t stream,
                              void* const* d_in, const int* in_sizes, int n_in,
                              void* d_out, int out_size,
                              const int* in_dtypes, int out_dtype,
                              void* const* d_out_all,
                              void* const* d_ws_all, size_t ws_bytes) {
  if (nranks != NR || n_in != 4 || (unsigned)rank >= (unsigned)NR || ws_bytes < WS_NEED ||
      in_sizes[0] != MDIM * KSH || in_sizes[1] != KSH * NDIM || out_size != MDIM * NDIM) {
    fprintf(stderr, "kernel_launch: unexpected problem shape\n");
    return;
  }
  Params p;
  p.src[0] = (const char*)d_in[0];
  p.src[1] = (const char*)d_in[1];
  p.sx = (const float*)d_in[2];
  p.sw = (const float*)d_in[3];
  p.out = (float*)d_out;
  p.ws0 = (char*)d_ws_all[rank];
  p.perm = 0;
  for (int d = 1, lo = 1, hi = 4; d < NR; ++d) {
    const bool same = ((0x4d >> rank) & 1) == ((0x4d >> ((rank + d) % NR)) & 1);
    p.perm |= (unsigned)d << (4 * (same ? lo++ : hi++));
  }
  for (int i = 0; i < NR; ++i) p.dst[i] = (char*)d_ws_all[(rank + ((p.perm >> (4 * i)) & 7)) % NR];
  fused_ag_gemm<<<NWG, NTHR, 0, stream>>>(p);
  HIP_CHECK(hipGetLastError());
}
```

```cpp
#include <hip/hip_runtime.h>
#include <rccl/rccl.h>
#include <cstdio>
#include <utility>

#define HIP_CHECK(expr) do { hipError_t _e = (expr); if (_e != hipSuccess) { \
    fprintf(stderr, "HIP error %s at %s:%d\n", hipGetErrorString(_e), __FILE__, __LINE__); \
    return; } } while (0)

namespace {

typedef unsigned u32;
typedef unsigned long long u64;
typedef u32 u32x2 __attribute__((ext_vector_type(2)));
typedef u32 u32x4 __attribute__((ext_vector_type(4)));
typedef int i32x4 __attribute__((ext_vector_type(4)));
typedef int i32x16 __attribute__((ext_vector_type(16)));
typedef float f32x4 __attribute__((ext_vector_type(4)));
#define LAS __attribute__((address_space(3)))
#define GAS __attribute__((address_space(1)))

constexpr int NR = 8;
constexpr int MDIM = 4096, NDIM = 4096, KSH = 512;
constexpr int NWG = 256, NTHR = 768;
constexpr u32 NQ = 64;
constexpr u32 UNIT = 16384;
constexpr u32 BLK_BYTES = NQ * UNIT;
constexpr u32 PAR_BYTES = 32 * BLK_BYTES;
constexpr size_t WS_CTR = 0;
constexpr size_t WS_TMO = 16384;
constexpr size_t WS_CLAIM = 20480;
constexpr size_t WS_TICK = 24576;
constexpr size_t WS_FLAG = 32768;
constexpr size_t WS_DBG = 131072;
constexpr size_t WS_TRC = 524288;
constexpr size_t WS_DATA = 1u << 20;
constexpr size_t WS_NEED = WS_DATA + 2ull * PAR_BYTES;
constexpr u64 DEADLINE = 40000000ull;
constexpr u32 SPIN_LIMIT = 10000000u;
constexpr u32 NPUSH = 48, PPL = NPUSH / NR;
constexpr u32 NCONS = 10;
constexpr u32 NSLOT = 4;
constexpr u32 XUNIT = 5 * 4096;
constexpr u32 L_X = NSLOT * UNIT, RING = L_X + NSLOT * XUNIT;
constexpr u32 L_SYNC = RING, L_ABORT = RING + 192, L_DST = RING + 196, L_TOK = RING + 200, L_ORD = RING + 224;
constexpr u32 LDS_BYTES = RING + 256;
#define PUSH_D 8

struct Params {
  const char* src[2];
  const float* sx;
  const float* sw;
  float* out;
  char* ws0;
  char* dst[NR];
  unsigned perm;
};
#define DBG(bit) 0
#define DEPTH ((u32)PUSH_D)
#define ROT 0
#define PFD 2
#define FIRSTW true
#define STATIC false
#define STAMP(idx) do {} while (0)
#define TSARG
#define TSPASS
#define TS(i) do {} while (0)

template <int N>
__device__ __forceinline__ void wait_vm() { asm volatile("s_waitcnt vmcnt(%c0)" ::"n"(N) : "memory"); }
__device__ __forceinline__ void wait_lgkm0() { asm volatile("s_waitcnt lgkmcnt(0)" ::: "memory"); }
__device__ __forceinline__ u64 rt() { return __builtin_amdgcn_s_memrealtime(); }
__device__ __forceinline__ int lane_id() {
  int ln;
  asm volatile("v_mbcnt_lo_u32_b32 %0, -1, 0\n v_mbcnt_hi_u32_b32 %0, -1, %0" : "=v"(ln));
  __builtin_assume(ln >= 0 && ln < 64);
  return ln;
}
__device__ __forceinline__ u32 rfl(u32 v) { return __builtin_amdgcn_readfirstlane(v); }
__device__ __forceinline__ __amdgpu_buffer_rsrc_t mkrsrc(const void* p, u32 bytes) {
  return __builtin_amdgcn_make_buffer_rsrc((void*)p, 0, (int)bytes, 0x00020000);
}
#define PIN() do { asm volatile("" ::: "memory"); __builtin_amdgcn_sched_barrier(0); } while (0)

__device__ __forceinline__ void load_unit(u32x4 (&R)[16], const char* x, const char* w, int lane, bool isw, u32 k) {
  const u32 s = k >> 4, blk = k & 15;
  const __amdgpu_buffer_rsrc_t rs = mkrsrc(isw ? w : x, (u32)MDIM * KSH);
  const u32 lw = (u32)(lane >> 4) * 16 * NDIM + (lane & 15) * 16, lx = (lane >> 2) * KSH + ((lane ^ (lane >> 4)) & 3) * 16;
  const u32 lo = (isw ? lw : lx) + (isw ? s * 64 * NDIM + blk * 256 : blk * 256 * KSH + s * 64);
  const u32 st = isw ? NDIM : 16 * KSH;
#pragma unroll
  for (int i = 0; i < 16; ++i) R[i] = __builtin_amdgcn_raw_buffer_load_b128(rs, lo, i * st, 0);
}

__device__ __forceinline__ void transpose16(u32x4 (&T)[16], const u32x4 (&R)[16]) {
#pragma unroll
  for (int d = 0; d < 4; ++d)
#pragma unroll
    for (int e = 0; e < 4; ++e) {
      const u32 a0 = R[4 * e][d], a1 = R[4 * e + 1][d], a2 = R[4 * e + 2][d], a3 = R[4 * e + 3][d];
      const u32 l01 = __builtin_amdgcn_perm(a1, a0, 0x05010400u), h01 = __builtin_amdgcn_perm(a1, a0, 0x07030602u);
      const u32 l23 = __builtin_amdgcn_perm(a3, a2, 0x05010400u), h23 = __builtin_amdgcn_perm(a3, a2, 0x07030602u);
      T[4 * d + 0][e] = __builtin_amdgcn_perm(l23, l01, 0x05040100u);
      T[4 * d + 1][e] = __builtin_amdgcn_perm(l23, l01, 0x07060302u);
      T[4 * d + 2][e] = __builtin_amdgcn_perm(h23, h01, 0x05040100u);
      T[4 * d + 3][e] = __builtin_amdgcn_perm(h23, h01, 0x07060302u);
    }
}

__device__ __forceinline__ bool lds_wait_ge(volatile LAS u32* w, int need) {
  for (u32 sp = 0; (int)rfl(*w) < need; ++sp) {
    if (sp > SPIN_LIMIT) return false;
    __builtin_amdgcn_s_sleep(1);
  }
  return true;
}

__device__ __forceinline__ u32 claim_add(__amdgpu_buffer_rsrc_t rc, int lane, u32 slot, int v) {
  return __builtin_amdgcn_raw_ptr_buffer_atomic_add_i32(v, rc, lane * 512 + slot * 64, 0, 0);
}

__device__ __forceinline__ void pusher(const Params& p, LAS char* L, int lane, u32 wave, u32 pref, const char* x, const char* w,
                                       char* ws0, char* dws, u32 perm TSARG) {
  TS(1);
  const u32 b = blockIdx.x, j = b >> 3;
  volatile LAS u32* TOK = (volatile LAS u32*)(L + L_TOK);
  volatile LAS u32* TIK = (volatile LAS u32*)(L + L_TOK + 4);
  volatile LAS u32* DST = (volatile LAS u32*)(L + L_DST);
  const int PF = PFD;
  u32x4 R[16], T[16];
  GAS u32* ctr = (GAS u32*)(ws0 + WS_CTR) + b * 16 + wave;
  const __amdgpu_buffer_rsrc_t rc = mkrsrc(ws0 + WS_CLAIM + j * 512, 512);
  u32 old = 0;
  if (wave == 0) old = claim_add(rc, lane, pref, 1);
  u32 cv = __hip_atomic_load(ctr, __ATOMIC_RELAXED, __HIP_MEMORY_SCOPE_AGENT);
  PIN();
  if (wave == 0) load_unit(R, x, w, lane, FIRSTW, j);
  if (wave == 11) ((volatile LAS u32*)(L + RING))[lane] = 0;
  wait_lgkm0();
  __builtin_amdgcn_s_barrier();
  if (wave == 1 || wave == 2) load_unit(R, x, w, lane, FIRSTW, wave * PPL + j);
  PIN();
  asm volatile("" : "+v"(cv), "+v"(old));
  const u32 call = rfl(cv) + 1;
  u32 slot = pref;
  if (wave == 0) {
#pragma nounroll
    for (int k = 0; k < 7 && rfl(old) + 1 != call; ++k) {
      claim_add(rc, lane, slot, -1);
      slot = (slot + 1) & 7;
      old = claim_add(rc, lane, slot, 1);
    }
    *DST = slot + 1;
  } else {
    lds_wait_ge(DST, 1);
    slot = (rfl(*DST) - 1) & 7;
  }
  if (slot != pref) dws = p.dst[slot];
  const u32 delta = (perm >> (slot * 4)) & 7;
  const __amdgpu_buffer_rsrc_t rd = mkrsrc(dws + WS_DATA + (size_t)(call & 1) * PAR_BYTES, PAR_BYTES);
  const __amdgpu_buffer_rsrc_t rt_ = mkrsrc(ws0 + WS_TICK + (call & 1) * 512, 512);
  if (j == 0 && wave == 11 && lane == 0)
    __hip_atomic_store((GAS u32*)(ws0 + WS_TICK + (~call & 1) * 512) + slot * 16, 0u, __ATOMIC_RELAXED, __HIP_MEMORY_SCOPE_AGENT);
  GAS u32* flags = (GAS u32*)(dws + WS_FLAG) + delta;
  const u32 depth = delta ? DEPTH : 16;
  const bool skip = (DBG(0) && delta != 0) || (DBG(2) && delta == 0);
  for (u32 m = wave; !skip; m += 12) {
    u32 k, isw;
    if (m < 6) {
      isw = (m < 3) == FIRSTW;
      k = (m < 3 ? m : m - 3) * PPL + j;
    } else {
      u32 sp = 0, n = ~0u;
      for (;; ++sp) {
        const u32x2 v = *(volatile LAS u32x2*)TOK;
        const u32 tok = rfl(v[0]), tik = rfl(v[1]);
        if (tik == ~0u || sp > SPIN_LIMIT) break;
        if (tik + 6 == m && tok + PF + 1 >= m) {
          n = STATIC ? m * PPL + (j ^ (m & 1)) : rfl(claim_add(rt_, lane, slot, 1)) + 6 * PPL;
          break;
        }
        __builtin_amdgcn_s_sleep(1);
      }
      *TIK = n < 256 ? m - 5 : ~0u;
      if (n >= 256) break;
      isw = n & 1;
      k = n >> 1;
    }
    if (m > 2) {
      if (!lds_wait_ge(TOK, (int)m - PF)) break;
      load_unit(R, x, w, lane, isw, k);
    }
    if (isw) transpose16(T, R);
    else {
#pragma unroll
      for (int i = 0; i < 16; ++i) T[i] = R[i];
    }
#pragma unroll
    for (int i = 0; i < 16; ++i) asm volatile("" : "+v"(T[i]));
    const u32 row = isw * 16 + (k & 15), q = (k >> 4) * 8;
    const u32 so = lane * 16 + (q + delta) * UNIT + row * BLK_BYTES, sw = isw << 6;
    wait_vm<0>();
    if (!lds_wait_ge(TOK, (int)m)) break;
#pragma unroll
    for (int i = 0; i < 16; ++i)
      __builtin_amdgcn_raw_buffer_store_b128(T[i], rd, ((so ^ ((i >> 2) * sw)) + (i >> 2) * 4096) + (i & 3) * 1024, 0, 17);
    if (depth != 16) wait_vm<PUSH_D>();
    *TOK = m + 1;
    wait_vm<0>();
    if (lane == 0) __hip_atomic_store(flags + row * NQ + q, call, __ATOMIC_RELAXED, __HIP_MEMORY_SCOPE_SYSTEM);
  }
  if (lane == 0) __hip_atomic_store(ctr, call, __ATOMIC_RELAXED, __HIP_MEMORY_SCOPE_AGENT);
}

template <bool ISW>
__device__ __forceinline__ void loader(const Params& p, LAS char* L, int lane, u32 wave, u32 rg, u32 cb) {
  char* const ws0 = p.ws0;
  const u32 b = blockIdx.x;
  GAS u32* ctr = (GAS u32*)(ws0 + WS_CTR) + b * 16 + wave;
  const u32 cv = __hip_atomic_load(ctr, __ATOMIC_RELAXED, __HIP_MEMORY_SCOPE_AGENT);
  __builtin_amdgcn_s_barrier();
  const u32 call = rfl(cv) + 1;
  const __amdgpu_buffer_rsrc_t rs = mkrsrc(ws0 + WS_DATA + (size_t)(call & 1) * PAR_BYTES + (ISW ? 16 * BLK_BYTES : 0), 16 * BLK_BYTES);
  GAS u32* tmo = (GAS u32*)(ws0 + WS_TMO);
  const u32 cl = min((u32)lane, NCONS - 1);
  const u32 waddr = L_SYNC + cl * 16 + 8, paddr = L_SYNC + cl * 16 + (ISW ? 4 : 0);
  const u32 vo = lane * 16;
  u32 issued = 0, landed = 0, sp = 0;
#define LD4 "buffer_load_dwordx4 %[vo], %[rs], %[t] offen sc0 sc1 lds\n"             \
            "buffer_load_dwordx4 %[vo], %[rs], %[t] offen offset:1024 sc0 sc1 lds\n" \
            "buffer_load_dwordx4 %[vo], %[rs], %[t] offen offset:2048 sc0 sc1 lds\n" \
            "buffer_load_dwordx4 %[vo], %[rs], %[t] offen offset:3072 sc0 sc1 lds\n"
#define TARGET NQ
#define TRC(v, idx)
#define TRCOUT
#define TRCCLOB
  if constexpr (ISW) {
    const __amdgpu_buffer_rsrc_t rf = mkrsrc(ws0 + WS_FLAG, 32 * NQ * 4);
    const u32 fo = lane * 4;
    const u32 r0 = (rg * 5 >> 2) * NQ * 4, r1 = min((rg * 5 + 4) >> 2, 15u) * NQ * 4, r2 = (16 + cb) * NQ * 4;
    u64 pend = ~0ull, rdy = DBG(1) ? ~0ull : 0, t0 = 0;
    u32 polling = 0, code, f0 = 0, f1 = 0, f2 = 0;
#define NXT "s_add_u32 m0, m0, 0x1000\n s_add_u32 %[t], %[t], 0x1000\n"
#define WLAND(k) "s_waitcnt vmcnt(" #k "+3)\n s_cbranch_scc1 Lpub%=\n s_waitcnt vmcnt(" #k ")\n"
    for (;;) {
      const u32 target = TARGET;
      u32 st, su, vd, ve;
      u64 av;
      asm volatile(
          "s_mov_b32 %[code], 64\n"
          "Ltop%=:\n"
          "s_and_b64 %[av], %[rdy], %[pend]\n"
          "s_cbranch_scc0 Lneed%=\n"
          "Lgot%=:\n"
          "s_add_u32 %[t], %[la], 3\n"
          "s_cmp_lt_u32 %[is], %[t]\n"
          "s_cbranch_scc0 Lland%=\n"
          "ds_read_b32 %[vd], %[wa]\n"
          "s_ff1_i32_b64 %[u], %[av]\n"
          "s_lshl_b32 %[t], %[is], 8\n"
          "s_add_u32 %[t], %[t], %[u]\n"
          "s_add_u32 %[t], %[t], 0x100\n"
          "v_mov_b32 %[ve], %[t]\n"
          "s_and_b32 %[t], %[is], 3\n"
          "s_lshl_b32 m0, %[t], 14\n"
          "s_lshl_b32 %[t], %[t], 2\n"
          "s_waitcnt lgkmcnt(0)\n"
          "v_add_u32 %[vd], 4, %[vd]\n"
          "v_cmp_ge_u32 vcc, %[is], %[vd]\n"
          "s_cbranch_vccnz Lnoslot%=\n"
          "v_mov_b32 %[vd], %[t]\n"
          "v_add_u32 %[vd], %[ord], %[vd]\n"
          "ds_write_b32 %[vd], %[ve]\n"
          "s_bitset0_b64 %[pend], %[u]\n"
          "s_lshl_b32 %[u], %[u], 14\n"
          "s_add_u32 %[t], %[u], %[g0]\n"
          LD4 NXT LD4 NXT LD4 NXT LD4
          TRC(tb, "%[is]")
          "s_add_u32 %[is], %[is], 1\n"
          "s_branch Ltop%=\n"
          "Lneed%=:\n"
          "s_cmp_eq_u64 %[pend], 0\n"
          "s_cbranch_scc1 Lland%=\n"
          "s_cmp_lg_u32 %[po], 0\n"
          "s_cbranch_scc1 Lpolled%=\n"
          "Lpoll%=:\n"
          "buffer_load_dword %[f0], %[fo], %[rf], %[r0] offen sc0 sc1\n"
          "buffer_load_dword %[f1], %[fo], %[rf], %[r1] offen sc0 sc1\n"
          "buffer_load_dword %[f2], %[fo], %[rf], %[r2] offen sc0 sc1\n"
          "s_mov_b32 %[po], 1\n"
          "Lpolled%=:\n"
          "s_cmp_lt_u32 %[la], %[is]\n"
          "s_cbranch_scc1 Lland%=\n"
          "s_waitcnt vmcnt(0)\n"
          "s_mov_b32 %[po], 0\n"
          "v_min3_u32 %[f0], %[f0], %[f1], %[f2]\n"
          "v_cmp_le_u32 %[rdy], %[call], %[f0]\n"
          "s_nop 0\n"
          "s_and_b64 %[av], %[rdy], %[pend]\n"
          "s_cbranch_scc1 Lgot%=\n"
          "s_sub_u32 %[code], %[code], 1\n"
          "s_cmp_eq_u32 %[code], 0\n"
          "s_cbranch_scc1 Lend%=\n"
          "s_sleep 2\n"
          "s_branch Lpoll%=\n"
          "Lnoslot%=:\n"
          "s_cmp_lt_u32 %[la], %[is]\n"
          "s_cbranch_scc1 Lland%=\n"
          "s_add_u32 %[sp], %[sp], 1\n"
          "s_cmp_gt_u32 %[sp], %[lim]\n"
          "s_cbranch_scc1 Lend%=\n"
          "s_sleep 1\n"
          "s_branch Ltop%=\n"
          "Lland%=:\n"
          "s_sub_u32 %[t], %[is], %[la]\n"
          "s_add_u32 %[la], %[la], 1\n"
          "v_mov_b32 %[vd], %[la]\n"
          "s_cmp_eq_u32 %[t], 1\n"
          "s_cbranch_scc1 Lw1%=\n"
          "s_cmp_eq_u32 %[t], 2\n"
          "s_cbranch_scc1 Lw2%=\n"
          "s_cmp_lg_u32 %[po], 0\n"
          WLAND(32)
          "s_branch Lpub%=\n"
          "Lw2%=:\n"
          "s_cmp_lg_u32 %[po], 0\n"
          WLAND(16)
          "s_branch Lpub%=\n"
          "Lw1%=:\n"
          "s_cmp_lg_u32 %[po], 0\n"
          WLAND(0)
          "Lpub%=:\n"
          "ds_write_b32 %[pa], %[vd]\n"
          "s_cmp_lt_u32 %[la], %[tg]\n"
          "s_cbranch_scc1 Ltop%=\n"
          "s_mov_b32 %[code], 200\n"
          "Lend%=:\n"
          : [rdy] "+s"(rdy), [pend] "+s"(pend), [is] "+s"(issued), [la] "+s"(landed), [po] "+s"(polling), [sp] "+s"(sp),
            [code] "=&s"(code), [t] "=&s"(st), [u] "=&s"(su), [av] "=&s"(av), [vd] "=&v"(vd), [ve] "=&v"(ve),
            [f0] "+{v100}"(f0), [f1] "+{v101}"(f1), [f2] "+{v102}"(f2) TRCOUT
          : [vo] "v"(vo), [wa] "v"(waddr), [pa] "v"(paddr), [fo] "v"(fo), [rs] "s"(rs), [rf] "s"(rf), [g0] "s"(cb * BLK_BYTES),
            [r0] "s"(r0), [r1] "s"(r1), [r2] "s"(r2), [call] "s"(call), [lim] "s"(SPIN_LIMIT), [tg] "s"(target), [ord] "n"(L_ORD)
          : "memory", "scc", "vcc" TRCCLOB);
      if (landed >= NQ || sp > SPIN_LIMIT) break;
      const u64 t = rt();
      if (t0 == 0) t0 = t;
      const u32 tv = rfl(__hip_atomic_load(tmo, __ATOMIC_RELAXED, __HIP_MEMORY_SCOPE_AGENT));
      if (tv == call || t - t0 > DEADLINE) break;
    }
#undef NXT
  } else {
    u32 goff[5];
#pragma unroll
    for (int g = 0; g < 5; ++g) {
      const u32 strip = min(rg * 5 + g, 63u);
      goff[g] = (strip >> 2) * BLK_BYTES + (strip & 3) * 4096;
    }
#define NXT(g) "s_add_u32 m0, m0, 0x1000\n s_add_u32 %[t], %[u], %[" #g "]\n"
    u32 st, su, vd;
    asm volatile(
        "Ltop%=:\n"
        "s_add_u32 %[t], %[la], 3\n"
        "s_min_u32 %[t], %[t], 64\n"
        "s_cmp_lt_u32 %[is], %[t]\n"
        "s_cbranch_scc0 Lland%=\n"
        "s_and_b32 %[t], %[is], 3\n"
        "s_lshl_b32 %[u], %[t], 2\n"
        "v_mov_b32 %[vd], %[u]\n"
        "v_add_u32 %[vd], %[ord], %[vd]\n"
        "ds_read_b32 %[vd], %[vd]\n"
        "s_mul_i32 %[t], %[t], %[stride]\n"
        "s_add_u32 m0, %[t], %[base]\n"
        "s_waitcnt lgkmcnt(0)\n"
        "v_readfirstlane_b32 %[u], %[vd]\n"
        "s_lshr_b32 %[t], %[u], 8\n"
        "s_cmp_gt_u32 %[t], %[is]\n"
        "s_cbranch_scc0 Lno%=\n"
        "s_cmp_gt_u32 %[t], 64\n"
        "s_cbranch_scc1 Lend%=\n"
        "s_and_b32 %[u], %[u], 63\n"
        "s_lshl_b32 %[u], %[u], 14\n"
        "s_add_u32 %[t], %[u], %[g0]\n"
        LD4 NXT(g1) LD4 NXT(g2) LD4 NXT(g3) LD4 NXT(g4) LD4
        TRC(tb, "%[is]")
        "s_add_u32 %[is], %[is], 1\n"
        "s_branch Ltop%=\n"
        "Lno%=:\n"
        "s_cmp_lt_u32 %[la], %[is]\n"
        "s_cbranch_scc1 Lland%=\n"
        "s_add_u32 %[sp], %[sp], 1\n"
        "s_cmp_gt_u32 %[sp], %[lim]\n"
        "s_cbranch_scc1 Lend%=\n"
        "s_sleep 1\n"
        "s_branch Ltop%=\n"
        "Lland%=:\n"
        "s_sub_u32 %[t], %[is], %[la]\n"
        "s_add_u32 %[la], %[la], 1\n"
        "v_mov_b32 %[vd], %[la]\n"
        "s_waitcnt vmcnt(40)\n"
        "s_cmp_eq_u32 %[t], 3\n"
        "s_cbranch_scc1 Lpub%=\n"
        "s_waitcnt vmcnt(20)\n"
        "s_cmp_eq_u32 %[t], 2\n"
        "s_cbranch_scc1 Lpub%=\n"
        "s_waitcnt vmcnt(0)\n"
        "Lpub%=:\n"
        "ds_write_b32 %[pa], %[vd]\n"
        "s_cmp_lt_u32 %[la], 64\n"
        "s_cbranch_scc1 Ltop%=\n"
        "Lend%=:\n"
        : [is] "+s"(issued), [la] "+s"(landed), [sp] "+s"(sp), [t] "=&s"(st), [u] "=&s"(su), [vd] "=&v"(vd) TRCOUT
        : [vo] "v"(vo), [pa] "v"(paddr), [rs] "s"(rs), [g0] "s"(goff[0]), [g1] "s"(goff[1]), [g2] "s"(goff[2]),
          [g3] "s"(goff[3]), [g4] "s"(goff[4]), [lim] "s"(SPIN_LIMIT), [stride] "n"(XUNIT), [base] "n"(L_X), [ord] "n"(L_ORD)
        : "memory", "scc" TRCCLOB);
#undef NXT
  }
  if (landed < NQ) {
    *(volatile LAS u32*)(L + L_ABORT) = 1;
    *(volatile LAS u32*)(L + L_ORD + (lane & 3) * 4) = ~0u;
    wait_lgkm0();
    *(volatile LAS u32*)(L + L_SYNC + cl * 16) = NQ;
    *(volatile LAS u32*)(L + L_SYNC + cl * 16 + 4) = NQ;
    if (lane == 0) __hip_atomic_store(tmo, call, __ATOMIC_RELAXED, __HIP_MEMORY_SCOPE_AGENT);
  }
  if (lane == 0) __hip_atomic_store(ctr, call, __ATOMIC_RELAXED, __HIP_MEMORY_SCOPE_AGENT);
  STAMP(ISW ? 6 : 5);
}

__global__ void __launch_bounds__(NTHR) fused_ag_gemm(Params p) {
  __shared__ __attribute__((aligned(1024))) char lds[LDS_BYTES];
  LAS char* L = (LAS char*)lds;
  volatile LAS u32* ABORT = (volatile LAS u32*)(L + L_ABORT);
  const int wave = rfl(threadIdx.x >> 6);
  const u32 b = blockIdx.x;
  char* const ws0 = p.ws0;

  if (b < NPUSH) {
    const u32 pref = (__builtin_amdgcn_s_getreg((3 << 11) | 20) + ROT) & 7;
    const char *x = p.src[0], *w = p.src[1];
    char *own = p.ws0, *dws = p.dst[pref];
    u32 perm = p.perm;
    asm volatile("" : "+s"(x), "+s"(w), "+s"(own), "+s"(dws), "+s"(perm));
    pusher(p, L, lane_id(), wave, pref, x, w, own, dws, perm TSPASS);
    return;
  }

  const u32 ti = (b >> 3) - PPL;
  const u32 rg = ti < 24 ? (ti >> 2) * 2 + (b & 1) : 12;
  const u32 cb = (ti < 24 ? ti & 3 : (b & 1) * 2 + ti - 24) * 4 + ((b & 7) >> 1);

  if (wave >= (int)NCONS) {
    __builtin_amdgcn_s_setprio(2);
    if (wave == (int)NCONS) loader<false>(p, L, lane_id(), wave, rg, cb);
    else loader<true>(p, L, lane_id(), wave, rg, cb);
    return;
  }

  {
    int lane = lane_id();
    if (wave == 0) ((volatile LAS u32*)(L + RING))[lane] = 0;
    wait_lgkm0();
    __builtin_amdgcn_s_barrier();
    if (wave == 0) STAMP(0);
    const int wr = wave >> 1, wc = wave & 1;
    u32 aaddr, aaddr1, baddr;
    {
      const int h = lane >> 5, c = lane & 31;
      aaddr = L_X + wr * 4096 + c * 64 + ((h ^ (c >> 2)) & 3) * 16;
      aaddr1 = aaddr ^ 32;
      baddr = (c & 3) * 4096 + h * 256 + (((8 * wc + (c >> 2)) ^ ((c & 3) << 2)) * 16);
    }
    const u32 saddr = L_SYNC + wave * 16;
    const __amdgpu_buffer_rsrc_t ro = mkrsrc(p.out, (u32)MDIM * NDIM * 4);
    const u32 so = (rg * 320 + wr * 64) * (NDIM * 4) + (cb * 256 + wc * 128) * 4;
    u32 touch = __builtin_amdgcn_raw_buffer_load_b32(ro, so + lane * (NDIM * 4), 0, 0);
    float scale = __builtin_bit_cast(float, rfl(__builtin_bit_cast(u32, p.sx[0] * p.sw[0])));
    asm volatile("" : "+s"(scale) : "v"(touch));
    i32x16 acc[2][4];
#define MM(t, A, B) "v_mfma_i32_32x32x32_i8 v[" #t "*16:" #t "*16+15], v[" #A ":" #A "+3], v[" #B ":" #B "+3], v[" #t "*16:" #t "*16+15]\n" \
                    "v_mfma_i32_32x32x32_i8 v[" #t "*16+64:" #t "*16+79], v[" #A "+4:" #A "+7], v[" #B ":" #B "+3], v[" #t "*16+64:" #t "*16+79]\n"
#define RDA(A, adr) "ds_read_b128 v[" #A ":" #A "+3], v" #adr "\n" \
                    "ds_read_b128 v[" #A "+4:" #A "+7], v" #adr " offset:2048\n"
#define RDB(t, kh) "ds_read_b128 v[144+4*" #t ":147+4*" #t "], v161 offset:" #t "*1024+" #kh "*512\n"
#define WL(n) "s_waitcnt lgkmcnt(" #n ")\n"
    u32 sq, sav, ssp;
    asm volatile(
        "v_mov_b32 v128, 0\n v_mov_b32 v129, 0\n v_mov_b32 v130, 0\n v_mov_b32 v131, 0\n"
        "s_mov_b32 %[q], 0\n"
        "s_mov_b32 %[sp], 0\n"
        "s_setprio 1\n"
        "s_nop 1\n"
        "v_mfma_i32_32x32x32_i8 v[0:15], v[128:131], v[128:131], 0\n"
        "v_mfma_i32_32x32x32_i8 v[16:31], v[128:131], v[128:131], 0\n"
        "v_mfma_i32_32x32x32_i8 v[32:47], v[128:131], v[128:131], 0\n"
        "v_mfma_i32_32x32x32_i8 v[48:63], v[128:131], v[128:131], 0\n"
        "v_mfma_i32_32x32x32_i8 v[64:79], v[128:131], v[128:131], 0\n"
        "v_mfma_i32_32x32x32_i8 v[80:95], v[128:131], v[128:131], 0\n"
        "v_mfma_i32_32x32x32_i8 v[96:111], v[128:131], v[128:131], 0\n"
        "v_mfma_i32_32x32x32_i8 v[112:127], v[128:131], v[128:131], 0\n"
        "Lspin%=:\n"
        "ds_read_b64 v[164:165], v162\n"
        WL(0)
        "v_min_u32 v164, v164, v165\n"
        "s_nop 0\n"
        "v_readfirstlane_b32 %[av], v164\n"
        "s_cmp_gt_u32 %[av], %[q]\n"
        "s_cbranch_scc1 Lgo%=\n"
        "s_add_u32 %[sp], %[sp], 1\n"
        "s_cmp_gt_u32 %[sp], %[lim]\n"
        "s_cbranch_scc1 Lend%=\n"
        "s_sleep 1\n"
        "s_branch Lspin%=\n"
        "Lgo%=:\n"
        RDA(128, 160) RDB(0, 0) RDB(1, 0) RDB(2, 0)
        "Lstep%=:\n"
        RDB(3, 0) RDA(136, 166)
        "s_add_u32 %[q], %[q], 1\n"
        "v_mov_b32 v163, %[q]\n"
        WL(5) MM(0, 128, 144)
        RDB(0, 1) WL(5) MM(1, 128, 148)
        RDB(1, 1) WL(5) MM(2, 128, 152)
        RDB(2, 1) "ds_read_b64 v[164:165], v162\n" WL(6) MM(3, 128, 156)
        RDB(3, 1) "ds_write_b32 v162, v163 offset:8\n"
        "s_cmp_lg_u32 %[q], %[lastq]\n"
        "s_cbranch_scc1 Lkeep%=\n"
        "s_setprio 0\n"
        "Lkeep%=:\n"
        "s_and_b32 %[av], %[q], 3\n"
        "s_cselect_b32 %[av], 0x5000, %[wrap]\n"
        "v_add_u32 v161, 0x4000, v161\n"
        "v_add_u32 v160, %[av], v160\n"
        "v_add_u32 v166, %[av], v166\n"
        "v_and_b32 v161, 0xffff, v161\n"
        WL(5) MM(0, 136, 144)
        WL(2)
        "v_min_u32 v164, v164, v165\n"
        "s_nop 0\n"
        "v_readfirstlane_b32 %[av], v164\n"
        "s_cmp_gt_u32 %[av], %[q]\n"
        "s_cbranch_scc0 Lslow%=\n"
        RDA(128, 160) RDB(0, 0) MM(1, 136, 148)
        RDB(1, 0) MM(2, 136, 152)
        RDB(2, 0) WL(6) MM(3, 136, 156)
        "s_branch Lstep%=\n"
        "Lslow%=:\n"
        MM(1, 136, 148) MM(2, 136, 152) WL(0) MM(3, 136, 156)
        "s_cmp_lt_u32 %[q], 64\n"
        "s_cbranch_scc1 Lspin%=\n"
        "Lend%=:\n"
        "s_nop 15\n s_nop 7\n"
        : "={v[0:15]}"(acc[0][0]), "={v[16:31]}"(acc[0][1]), "={v[32:47]}"(acc[0][2]), "={v[48:63]}"(acc[0][3]),
          "={v[64:79]}"(acc[1][0]), "={v[80:95]}"(acc[1][1]), "={v[96:111]}"(acc[1][2]), "={v[112:127]}"(acc[1][3]),
          "+{v160}"(aaddr), "+{v166}"(aaddr1), "+{v161}"(baddr), [q] "=&s"(sq), [av] "=&s"(sav), [sp] "=&s"(ssp)
        : "{v162}"(saddr), [lim] "s"(SPIN_LIMIT), [wrap] "s"(0u - 3 * XUNIT), [lastq] "s"(DBG(7) ? 100u : NQ - NSLOT)
        : "memory", "scc", "v128", "v129", "v130", "v131", "v132", "v133", "v134", "v135", "v136", "v137", "v138", "v139", "v140",
          "v141", "v142", "v143", "v144", "v145", "v146", "v147", "v148", "v149", "v150", "v151", "v152", "v153", "v154", "v155",
          "v156", "v157", "v158", "v159", "v163", "v164", "v165" TRCCLOB);
    lane = lane_id();
    if (wave == 0) STAMP(2);
    if (rfl(ABORT[0]) != 0 || sq != NQ) scale = __builtin_nanf("");
    if (rg * 5 + wr >= 64) return;
    const u32 vo = so + (lane >> 5) * 4 * (NDIM * 4) + (lane & 31) * 16;
#pragma unroll
    for (int m = 0; m < 2; ++m)
#pragma unroll
      for (int r = 0; r < 16; ++r) {
        f32x4 v;
#pragma unroll
        for (int t = 0; t < 4; ++t) v[t] = __builtin_elementwise_maximum((float)acc[m][t][r] * scale, 0.f);
        __builtin_amdgcn_raw_buffer_store_b128(__builtin_bit_cast(u32x4, v), ro,
                                               vo + (m * 32 + (r & 3) + 8 * (r >> 2)) * (NDIM * 4), 0, 16);
      }
  }
}

}

extern "C" void kernel_launch(int rank, int nranks, ncclComm_t comm,
                              hipStream_t stream,
                              void* const* d_in, const int* in_sizes, int n_in,
                              void* d_out, int out_size,
                              const int* in_dtypes, int out_dtype,
                              void* const* d_out_all,
                              void* const* d_ws_all, size_t ws_bytes) {
  if (nranks != NR || n_in != 4 || (unsigned)rank >= (unsigned)NR || ws_bytes < WS_NEED ||
      in_sizes[0] != MDIM * KSH || in_sizes[1] != KSH * NDIM || out_size != MDIM * NDIM) {
    fprintf(stderr, "kernel_launch: unexpected problem shape\n");
    return;
  }
  Params p;
  p.src[0] = (const char*)d_in[0];
  p.src[1] = (const char*)d_in[1];
  p.sx = (const float*)d_in[2];
  p.sw = (const float*)d_in[3];
  p.out = (float*)d_out;
  p.ws0 = (char*)d_ws_all[rank];
  p.perm = 0;
  for (int d = 1, lo = 1, hi = 4; d < NR; ++d) {
    const bool same = ((0x4d >> rank) & 1) == ((0x4d >> ((rank + d) % NR)) & 1);
    p.perm |= (unsigned)d << (4 * (same ? lo++ : hi++));
  }
  for (int i = 0; i < NR; ++i) p.dst[i] = (char*)d_ws_all[(rank + ((p.perm >> (4 * i)) & 7)) % NR];
  fused_ag_gemm<<<NWG, NTHR, 0, stream>>>(p);
  HIP_CHECK(hipGetLastError());
}
```

```cpp
#include <hip/hip_runtime.h>
#include <rccl/rccl.h>
#include <cstdio>
#include <utility>

#define HIP_CHECK(expr) do { hipError_t _e = (expr); if (_e != hipSuccess) { \
    fprintf(stderr, "HIP error %s at %s:%d\n", hipGetErrorString(_e), __FILE__, __LINE__); \
    return; } } while (0)

namespace {

typedef unsigned u32;
typedef unsigned long long u64;
typedef u32 u32x2 __attribute__((ext_vector_type(2)));
typedef u32 u32x4 __attribute__((ext_vector_type(4)));
typedef int i32x4 __attribute__((ext_vector_type(4)));
typedef int i32x16 __attribute__((ext_vector_type(16)));
typedef float f32x4 __attribute__((ext_vector_type(4)));
#define LAS __attribute__((address_space(3)))
#define GAS __attribute__((address_space(1)))

constexpr int NR = 8;
constexpr int MDIM = 4096, NDIM = 4096, KSH = 512;
constexpr int NWG = 256, NTHR = 768;
constexpr u32 NQ = 64;
constexpr u32 UNIT = 16384;
constexpr u32 BLK_BYTES = NQ * UNIT;
constexpr u32 PAR_BYTES = 32 * BLK_BYTES;
constexpr size_t WS_CTR = 0;
constexpr size_t WS_TMO = 16384;
constexpr size_t WS_CLAIM = 20480;
constexpr size_t WS_TICK = 24576;
constexpr size_t WS_FLAG = 32768;
constexpr size_t WS_DBG = 131072;
constexpr size_t WS_TRC = 524288;
constexpr size_t WS_DATA = 1u << 20;
constexpr size_t WS_NEED = WS_DATA + 2ull * PAR_BYTES;
constexpr u64 DEADLINE = 40000000ull;
constexpr u32 SPIN_LIMIT = 10000000u;
constexpr u32 NPUSH = 48, PPL = NPUSH / NR;
constexpr u32 NCONS = 10;
constexpr u32 NSLOT = 4;
constexpr u32 XUNIT = 5 * 4096;
constexpr u32 L_X = NSLOT * UNIT, RING = L_X + NSLOT * XUNIT;
constexpr u32 L_SYNC = RING, L_ABORT = RING + 192, L_DST = RING + 196, L_TOK = RING + 200, L_ORD = RING + 224;
constexpr u32 LDS_BYTES = RING + 256;
#define PUSH_D 8

struct Params {
  const char* src[2];
  const float* sx;
  const float* sw;
  float* out;
  char* ws0;
  char* dst[NR];
  unsigned perm;
};
#define DBG(bit) 0
#define DEPTH ((u32)PUSH_D)
#define ROT 0
#define PFD 2
#define FIRSTW false
#define STATIC false
#define OPT(bit) ((bit) == 9)
#define STAMP(idx) do {} while (0)
#define TSARG
#define TSPASS
#define TS(i) do {} while (0)

template <int N>
__device__ __forceinline__ void wait_vm() { asm volatile("s_waitcnt vmcnt(%c0)" ::"n"(N) : "memory"); }
__device__ __forceinline__ void wait_lgkm0() { asm volatile("s_waitcnt lgkmcnt(0)" ::: "memory"); }
__device__ __forceinline__ u64 rt() { return __builtin_amdgcn_s_memrealtime(); }
__device__ __forceinline__ int lane_id() {
  int ln;
  asm volatile("v_mbcnt_lo_u32_b32 %0, -1, 0\n v_mbcnt_hi_u32_b32 %0, -1, %0" : "=v"(ln));
  __builtin_assume(ln >= 0 && ln < 64);
  return ln;
}
__device__ __forceinline__ u32 rfl(u32 v) { return __builtin_amdgcn_readfirstlane(v); }
__device__ __forceinline__ __amdgpu_buffer_rsrc_t mkrsrc(const void* p, u32 bytes) {
  return __builtin_amdgcn_make_buffer_rsrc((void*)p, 0, (int)bytes, 0x00020000);
}
#define PIN() do { asm volatile("" ::: "memory"); __builtin_amdgcn_sched_barrier(0); } while (0)

__device__ __forceinline__ void load_unit(u32x4 (&R)[16], const char* x, const char* w, int lane, bool isw, u32 k) {
  const u32 s = k >> 4, blk = k & 15;
  const __amdgpu_buffer_rsrc_t rs = mkrsrc(isw ? w : x, (u32)MDIM * KSH);
  const u32 lw = (u32)(lane >> 4) * 16 * NDIM + (lane & 15) * 16, lx = (lane >> 2) * KSH + ((lane ^ (lane >> 4)) & 3) * 16;
  const u32 lo = (isw ? lw : lx) + (isw ? s * 64 * NDIM + blk * 256 : blk * 256 * KSH + s * 64);
  const u32 st = rfl(isw ? NDIM : 16 * KSH);
#pragma unroll
  for (int i = 0; i < 16; ++i) R[i] = __builtin_amdgcn_raw_buffer_load_b128(rs, lo, i * st, 0);
}

__device__ __forceinline__ void transpose16(u32x4 (&T)[16], const u32x4 (&R)[16]) {
#pragma unroll
  for (int d = 0; d < 4; ++d)
#pragma unroll
    for (int e = 0; e < 4; ++e) {
      const u32 a0 = R[4 * e][d], a1 = R[4 * e + 1][d], a2 = R[4 * e + 2][d], a3 = R[4 * e + 3][d];
      const u32 l01 = __builtin_amdgcn_perm(a1, a0, 0x05010400u), h01 = __builtin_amdgcn_perm(a1, a0, 0x07030602u);
      const u32 l23 = __builtin_amdgcn_perm(a3, a2, 0x05010400u), h23 = __builtin_amdgcn_perm(a3, a2, 0x07030602u);
      T[4 * d + 0][e] = __builtin_amdgcn_perm(l23, l01, 0x05040100u);
      T[4 * d + 1][e] = __builtin_amdgcn_perm(l23, l01, 0x07060302u);
      T[4 * d + 2][e] = __builtin_amdgcn_perm(h23, h01, 0x05040100u);
      T[4 * d + 3][e] = __builtin_amdgcn_perm(h23, h01, 0x07060302u);
    }
}

__device__ __forceinline__ bool lds_wait_ge(volatile LAS u32* w, int need) {
  for (u32 sp = 0; (int)rfl(*w) < need; ++sp) {
    if (sp > SPIN_LIMIT) return false;
    __builtin_amdgcn_s_sleep(1);
  }
  return true;
}

__device__ __forceinline__ u32 claim_add(__amdgpu_buffer_rsrc_t rc, int lane, u32 slot, int v) {
  return __builtin_amdgcn_raw_ptr_buffer_atomic_add_i32(v, rc, lane * 512 + slot * 64, 0, 0);
}

__device__ __forceinline__ void pusher(const Params& p, LAS char* L, int lane, u32 wave, u32 pref, const char* x, const char* w,
                                       char* ws0, char* dws, u32 perm TSARG) {
  TS(1);
  const u32 b = blockIdx.x, j = b >> 3;
  volatile LAS u32* TOK = (volatile LAS u32*)(L + L_TOK);
  volatile LAS u32* TIK = (volatile LAS u32*)(L + L_TOK + 4);
  volatile LAS u32* DST = (volatile LAS u32*)(L + L_DST);
  const int PF = PFD;
  u32x4 R[16], T[16];
  GAS u32* ctr = (GAS u32*)(ws0 + WS_CTR) + b * 16 + wave;
  const __amdgpu_buffer_rsrc_t rc = mkrsrc(ws0 + WS_CLAIM + j * 512, 512);
  u32 old = 0;
  if (wave == 0) old = claim_add(rc, lane, pref, 1);
  u32 cv = __hip_atomic_load(ctr, __ATOMIC_RELAXED, __HIP_MEMORY_SCOPE_AGENT);
  PIN();
  if (wave == 0) load_unit(R, x, w, lane, FIRSTW, j);
  if (wave == 11) ((volatile LAS u32*)(L + RING))[lane] = 0;
  wait_lgkm0();
  __builtin_amdgcn_s_barrier();
  if (wave == 1 || wave == 2) load_unit(R, x, w, lane, FIRSTW, wave * PPL + j);
  const __amdgpu_buffer_rsrc_t r0 = mkrsrc(dws + WS_DATA, 2 * PAR_BYTES);
  u32 so0 = lane * 16 + ((perm >> (pref * 4)) & 7) * UNIT + j * BLK_BYTES;
  asm volatile("" : "+v"(so0));
  PIN();
  asm volatile("" : "+v"(cv), "+v"(old));
  const u32 call = rfl(cv) + 1;
  const bool pre = wave == 0 && rfl(old) + 1 == call && !FIRSTW && !OPT(11);
  if (pre) {
#pragma unroll
    for (int i = 0; i < 16; ++i) __builtin_amdgcn_raw_buffer_store_b128(R[i], r0, so0 + (i & 3) * 1024, (call & 1) * PAR_BYTES + (i >> 2) * 4096, 17);
    PIN();
  }
  u32 slot = pref;
  if (wave == 0) {
    if (!pre) {
#pragma nounroll
      for (int k = 0; k < 7 && rfl(old) + 1 != call; ++k) {
        claim_add(rc, lane, slot, -1);
        slot = (slot + 1) & 7;
        old = claim_add(rc, lane, slot, 1);
      }
      asm volatile("" : "+v"(old));
    }
    *DST = slot + 1;
  } else {
    lds_wait_ge(DST, 1);
    slot = (rfl(*DST) - 1) & 7;
  }
  if (slot != pref) dws = p.dst[slot];
  const u32 delta = (perm >> (slot * 4)) & 7;
  const __amdgpu_buffer_rsrc_t rd = mkrsrc(dws + WS_DATA + (size_t)(call & 1) * PAR_BYTES, PAR_BYTES);
  const __amdgpu_buffer_rsrc_t rt_ = mkrsrc(ws0 + WS_TICK + (call & 1) * 512, 512);
  if (j == 0 && wave == 11 && lane == 0)
    __hip_atomic_store((GAS u32*)(ws0 + WS_TICK + (~call & 1) * 512) + slot * 16, 0u, __ATOMIC_RELAXED, __HIP_MEMORY_SCOPE_AGENT);
  GAS u32* flags = (GAS u32*)(dws + WS_FLAG) + delta;
  const u32 depth = delta ? DEPTH : 16;
  const bool skip = (DBG(0) && delta != 0) || (DBG(2) && delta == 0);
  for (u32 m = wave; !skip; m += 12) {
    u32 k, isw;
    if (m < 6) {
      isw = (m < 3) == FIRSTW;
      k = (m < 3 ? m : m - 3) * PPL + j;
    } else {
      u32 sp = 0, n = ~0u;
      for (;; ++sp) {
        const u32x2 v = *(volatile LAS u32x2*)TOK;
        const u32 tok = rfl(v[0]), tik = rfl(v[1]);
        if (tik == ~0u || sp > SPIN_LIMIT) break;
        if (tik + 6 == m && tok + PF + 1 >= m) {
          n = STATIC ? m * PPL + (j ^ (m & 1)) : rfl(claim_add(rt_, lane, slot, 1)) + 6 * PPL;
          break;
        }
        __builtin_amdgcn_s_sleep(1);
      }
      *TIK = n < 256 ? m - 5 : ~0u;
      if (n >= 256) break;
      isw = n & 1;
      k = n >> 1;
      if (OPT(9) && isw && k >= 32 && (k & 4)) k ^= 3;
    }
    isw = rfl(isw);
    k = rfl(k);
    if (m > 2) {
      if (!lds_wait_ge(TOK, (int)m - PF)) break;
      load_unit(R, x, w, lane, isw, k);
    }
    const u32 row = isw * 16 + (k & 15), q = (k >> 4) * 8;
    if (m != 0 || !pre) {
      if (isw) transpose16(T, R);
      else {
#pragma unroll
        for (int i = 0; i < 16; ++i) T[i] = R[i];
      }
#pragma unroll
      for (int i = 0; i < 16; ++i) asm volatile("" : "+v"(T[i]));
      const u32 so = lane * 16 + (q + delta) * UNIT + row * BLK_BYTES, sw = isw << 6;
      wait_vm<0>();
      if (!lds_wait_ge(TOK, (int)m)) break;
#pragma unroll
      for (int i = 0; i < 16; ++i)
        __builtin_amdgcn_raw_buffer_store_b128(T[i], rd, ((so ^ ((i >> 2) * sw)) + (i >> 2) * 4096) + (i & 3) * 1024, 0, 17);
    }
    if (depth != 16) wait_vm<PUSH_D>();
    *TOK = m + 1;
    wait_vm<0>();
    if (lane == 0) __hip_atomic_store(flags + row * NQ + q, call, __ATOMIC_RELAXED, __HIP_MEMORY_SCOPE_SYSTEM);
  }
  if (lane == 0) __hip_atomic_store(ctr, call, __ATOMIC_RELAXED, __HIP_MEMORY_SCOPE_AGENT);
}

template <bool ISW>
__device__ __forceinline__ void loader(const Params& p, LAS char* L, int lane, u32 wave, u32 rg, u32 cb) {
  char* const ws0 = p.ws0;
  const u32 b = blockIdx.x;
  GAS u32* ctr = (GAS u32*)(ws0 + WS_CTR) + b * 16 + wave;
  const u32 cv = __hip_atomic_load(ctr, __ATOMIC_RELAXED, __HIP_MEMORY_SCOPE_AGENT);
  __builtin_amdgcn_s_barrier();
  const u32 call = rfl(cv) + 1;
  const __amdgpu_buffer_rsrc_t rs = mkrsrc(ws0 + WS_DATA + (size_t)(call & 1) * PAR_BYTES + (ISW ? 16 * BLK_BYTES : 0), 16 * BLK_BYTES);
  GAS u32* tmo = (GAS u32*)(ws0 + WS_TMO);
  const u32 cl = min((u32)lane, NCONS - 1);
  const u32 waddr = L_SYNC + cl * 16 + 8, paddr = L_SYNC + cl * 16 + (ISW ? 4 : 0);
  const u32 vo = lane * 16;
  u32 issued = 0, landed = 0, sp = 0;
#define LD4 "buffer_load_dwordx4 %[vo], %[rs], %[t] offen sc0 sc1 lds\n"             \
            "buffer_load_dwordx4 %[vo], %[rs], %[t] offen offset:1024 sc0 sc1 lds\n" \
            "buffer_load_dwordx4 %[vo], %[rs], %[t] offen offset:2048 sc0 sc1 lds\n" \
            "buffer_load_dwordx4 %[vo], %[rs], %[t] offen offset:3072 sc0 sc1 lds\n"
#define TARGET NQ
#define TRC(v, idx)
#define TRCOUT
#define TRCCLOB
  if constexpr (ISW) {
    const __amdgpu_buffer_rsrc_t rf = mkrsrc(ws0 + WS_FLAG, 32 * NQ * 4);
    const u32 fo = lane * 4;
    const u32 r0 = (rg * 5 >> 2) * NQ * 4, r1 = min((rg * 5 + 4) >> 2, 15u) * NQ * 4, r2 = (16 + cb) * NQ * 4;
    u64 pend = ~0ull, rdy = DBG(1) ? ~0ull : 0, t0 = 0;
    u32 polling = 0, code, f0 = 0, f1 = 0, f2 = 0;
#define NXT "s_add_u32 m0, m0, 0x1000\n s_add_u32 %[t], %[t], 0x1000\n"
#define WLAND(k) "s_waitcnt vmcnt(" #k "+3)\n s_cbranch_scc1 Lpub%=\n s_waitcnt vmcnt(" #k ")\n"
    for (;;) {
      const u32 target = TARGET;
      u32 st, su, vd, ve;
      u64 av;
      asm volatile(
          "s_mov_b32 %[code], 64\n"
          "Ltop%=:\n"
          "s_and_b64 %[av], %[rdy], %[pend]\n"
          "s_cbranch_scc0 Lneed%=\n"
          "Lgot%=:\n"
          "s_add_u32 %[t], %[la], 3\n"
          "s_cmp_lt_u32 %[is], %[t]\n"
          "s_cbranch_scc0 Lland%=\n"
          "ds_read_b32 %[vd], %[wa]\n"
          "s_ff1_i32_b64 %[u], %[av]\n"
          "s_lshl_b32 %[t], %[is], 8\n"
          "s_add_u32 %[t], %[t], %[u]\n"
          "s_add_u32 %[t], %[t], 0x100\n"
          "v_mov_b32 %[ve], %[t]\n"
          "s_and_b32 %[t], %[is], 3\n"
          "s_lshl_b32 m0, %[t], 14\n"
          "s_lshl_b32 %[t], %[t], 2\n"
          "s_waitcnt lgkmcnt(0)\n"
          "v_add_u32 %[vd], 4, %[vd]\n"
          "v_cmp_ge_u32 vcc, %[is], %[vd]\n"
          "s_cbranch_vccnz Lnoslot%=\n"
          "v_mov_b32 %[vd], %[t]\n"
          "v_add_u32 %[vd], %[ord], %[vd]\n"
          "ds_write_b32 %[vd], %[ve]\n"
          "s_bitset0_b64 %[pend], %[u]\n"
          "s_lshl_b32 %[u], %[u], 14\n"
          "s_add_u32 %[t], %[u], %[g0]\n"
          LD4 NXT LD4 NXT LD4 NXT LD4
          TRC(tb, "%[is]")
          "s_add_u32 %[is], %[is], 1\n"
          "s_branch Ltop%=\n"
          "Lneed%=:\n"
          "s_cmp_eq_u64 %[pend], 0\n"
          "s_cbranch_scc1 Lland%=\n"
          "s_cmp_lg_u32 %[po], 0\n"
          "s_cbranch_scc1 Lpolled%=\n"
          "Lpoll%=:\n"
          "buffer_load_dword %[f0], %[fo], %[rf], %[r0] offen sc0 sc1\n"
          "buffer_load_dword %[f1], %[fo], %[rf], %[r1] offen sc0 sc1\n"
          "buffer_load_dword %[f2], %[fo], %[rf], %[r2] offen sc0 sc1\n"
          "s_mov_b32 %[po], 1\n"
          "Lpolled%=:\n"
          "s_cmp_lt_u32 %[la], %[is]\n"
          "s_cbranch_scc1 Lland%=\n"
          "s_waitcnt vmcnt(0)\n"
          "s_mov_b32 %[po], 0\n"
          "v_min3_u32 %[f0], %[f0], %[f1], %[f2]\n"
          "v_cmp_le_u32 %[rdy], %[call], %[f0]\n"
          "s_nop 0\n"
          "s_and_b64 %[av], %[rdy], %[pend]\n"
          "s_cbranch_scc1 Lgot%=\n"
          "s_sub_u32 %[code], %[code], 1\n"
          "s_cmp_eq_u32 %[code], 0\n"
          "s_cbranch_scc1 Lend%=\n"
          "s_sleep 2\n"
          "s_branch Lpoll%=\n"
          "Lnoslot%=:\n"
          "s_cmp_lt_u32 %[la], %[is]\n"
          "s_cbranch_scc1 Lland%=\n"
          "s_add_u32 %[sp], %[sp], 1\n"
          "s_cmp_gt_u32 %[sp], %[lim]\n"
          "s_cbranch_scc1 Lend%=\n"
          "s_sleep 1\n"
          "s_branch Ltop%=\n"
          "Lland%=:\n"
          "s_sub_u32 %[t], %[is], %[la]\n"
          "s_add_u32 %[la], %[la], 1\n"
          "v_mov_b32 %[vd], %[la]\n"
          "s_cmp_eq_u32 %[t], 1\n"
          "s_cbranch_scc1 Lw1%=\n"
          "s_cmp_eq_u32 %[t], 2\n"
          "s_cbranch_scc1 Lw2%=\n"
          "s_cmp_lg_u32 %[po], 0\n"
          WLAND(32)
          "s_branch Lpub%=\n"
          "Lw2%=:\n"
          "s_cmp_lg_u32 %[po], 0\n"
          WLAND(16)
          "s_branch Lpub%=\n"
          "Lw1%=:\n"
          "s_cmp_lg_u32 %[po], 0\n"
          WLAND(0)
          "Lpub%=:\n"
          "ds_write_b32 %[pa], %[vd]\n"
          "s_cmp_lt_u32 %[la], %[tg]\n"
          "s_cbranch_scc1 Ltop%=\n"
          "s_mov_b32 %[code], 200\n"
          "Lend%=:\n"
          : [rdy] "+s"(rdy), [pend] "+s"(pend), [is] "+s"(issued), [la] "+s"(landed), [po] "+s"(polling), [sp] "+s"(sp),
            [code] "=&s"(code), [t] "=&s"(st), [u] "=&s"(su), [av] "=&s"(av), [vd] "=&v"(vd), [ve] "=&v"(ve),
            [f0] "+{v100}"(f0), [f1] "+{v101}"(f1), [f2] "+{v102}"(f2) TRCOUT
          : [vo] "v"(vo), [wa] "v"(waddr), [pa] "v"(paddr), [fo] "v"(fo), [rs] "s"(rs), [rf] "s"(rf), [g0] "s"(cb * BLK_BYTES),
            [r0] "s"(r0), [r1] "s"(r1), [r2] "s"(r2), [call] "s"(call), [lim] "s"(SPIN_LIMIT), [tg] "s"(target), [ord] "n"(L_ORD)
          : "memory", "scc", "vcc" TRCCLOB);
      if (landed >= NQ || sp > SPIN_LIMIT) break;
      const u64 t = rt();
      if (t0 == 0) t0 = t;
      const u32 tv = rfl(__hip_atomic_load(tmo, __ATOMIC_RELAXED, __HIP_MEMORY_SCOPE_AGENT));
      if (tv == call || t - t0 > DEADLINE) break;
    }
#undef NXT
  } else {
    u32 goff[5];
#pragma unroll
    for (int g = 0; g < 5; ++g) {
      const u32 strip = min(rg * 5 + g, 63u);
      goff[g] = (strip >> 2) * BLK_BYTES + (strip & 3) * 4096;
    }
#define NXT(g) "s_add_u32 m0, m0, 0x1000\n s_add_u32 %[t], %[u], %[" #g "]\n"
    u32 st, su, vd;
    asm volatile(
        "Ltop%=:\n"
        "s_add_u32 %[t], %[la], 3\n"
        "s_min_u32 %[t], %[t], 64\n"
        "s_cmp_lt_u32 %[is], %[t]\n"
        "s_cbranch_scc0 Lland%=\n"
        "s_and_b32 %[t], %[is], 3\n"
        "s_lshl_b32 %[u], %[t], 2\n"
        "v_mov_b32 %[vd], %[u]\n"
        "v_add_u32 %[vd], %[ord], %[vd]\n"
        "ds_read_b32 %[vd], %[vd]\n"
        "s_mul_i32 %[t], %[t], %[stride]\n"
        "s_add_u32 m0, %[t], %[base]\n"
        "s_waitcnt lgkmcnt(0)\n"
        "v_readfirstlane_b32 %[u], %[vd]\n"
        "s_lshr_b32 %[t], %[u], 8\n"
        "s_cmp_gt_u32 %[t], %[is]\n"
        "s_cbranch_scc0 Lno%=\n"
        "s_cmp_gt_u32 %[t], 64\n"
        "s_cbranch_scc1 Lend%=\n"
        "s_and_b32 %[u], %[u], 63\n"
        "s_lshl_b32 %[u], %[u], 14\n"
        "s_add_u32 %[t], %[u], %[g0]\n"
        LD4 NXT(g1) LD4 NXT(g2) LD4 NXT(g3) LD4 NXT(g4) LD4
        TRC(tb, "%[is]")
        "s_add_u32 %[is], %[is], 1\n"
        "s_branch Ltop%=\n"
        "Lno%=:\n"
        "s_cmp_lt_u32 %[la], %[is]\n"
        "s_cbranch_scc1 Lland%=\n"
        "s_add_u32 %[sp], %[sp], 1\n"
        "s_cmp_gt_u32 %[sp], %[lim]\n"
        "s_cbranch_scc1 Lend%=\n"
        "s_sleep 1\n"
        "s_branch Ltop%=\n"
        "Lland%=:\n"
        "s_sub_u32 %[t], %[is], %[la]\n"
        "s_add_u32 %[la], %[la], 1\n"
        "v_mov_b32 %[vd], %[la]\n"
        "s_waitcnt vmcnt(40)\n"
        "s_cmp_eq_u32 %[t], 3\n"
        "s_cbranch_scc1 Lpub%=\n"
        "s_waitcnt vmcnt(20)\n"
        "s_cmp_eq_u32 %[t], 2\n"
        "s_cbranch_scc1 Lpub%=\n"
        "s_waitcnt vmcnt(0)\n"
        "Lpub%=:\n"
        "ds_write_b32 %[pa], %[vd]\n"
        "s_cmp_lt_u32 %[la], 64\n"
        "s_cbranch_scc1 Ltop%=\n"
        "Lend%=:\n"
        : [is] "+s"(issued), [la] "+s"(landed), [sp] "+s"(sp), [t] "=&s"(st), [u] "=&s"(su), [vd] "=&v"(vd) TRCOUT
        : [vo] "v"(vo), [pa] "v"(paddr), [rs] "s"(rs), [g0] "s"(goff[0]), [g1] "s"(goff[1]), [g2] "s"(goff[2]),
          [g3] "s"(goff[3]), [g4] "s"(goff[4]), [lim] "s"(SPIN_LIMIT), [stride] "n"(XUNIT), [base] "n"(L_X), [ord] "n"(L_ORD)
        : "memory", "scc" TRCCLOB);
#undef NXT
  }
  if (landed < NQ) {
    *(volatile LAS u32*)(L + L_ABORT) = 1;
    *(volatile LAS u32*)(L + L_ORD + (lane & 3) * 4) = ~0u;
    wait_lgkm0();
    *(volatile LAS u32*)(L + L_SYNC + cl * 16) = NQ;
    *(volatile LAS u32*)(L + L_SYNC + cl * 16 + 4) = NQ;
    if (lane == 0) __hip_atomic_store(tmo, call, __ATOMIC_RELAXED, __HIP_MEMORY_SCOPE_AGENT);
  }
  if (lane == 0) __hip_atomic_store(ctr, call, __ATOMIC_RELAXED, __HIP_MEMORY_SCOPE_AGENT);
  STAMP(ISW ? 6 : 5);
}

__global__ void __launch_bounds__(NTHR) fused_ag_gemm(Params p) {
  __shared__ __attribute__((aligned(1024))) char lds[LDS_BYTES];
  LAS char* L = (LAS char*)lds;
  volatile LAS u32* ABORT = (volatile LAS u32*)(L + L_ABORT);
  const int wave = rfl(threadIdx.x >> 6);
  const u32 b = blockIdx.x;
  char* const ws0 = p.ws0;

  if (__builtin_expect(b < NPUSH, 1)) {
    const u32 pref = (__builtin_amdgcn_s_getreg((3 << 11) | 20) + ROT) & 7;
    const char *x = p.src[0], *w = p.src[1];
    char *own = p.ws0, *dws = p.dst[pref];
    u32 perm = p.perm;
    asm volatile("" : "+s"(x), "+s"(w), "+s"(own), "+s"(dws), "+s"(perm));
    pusher(p, L, lane_id(), wave, pref, x, w, own, dws, perm TSPASS);
    return;
  }

  const u32 ti = (b >> 3) - PPL;
  const u32 rg = ti < 24 ? (ti >> 2) * 2 + (b & 1) : 12;
  const u32 cb = (ti < 24 ? ti & 3 : (b & 1) * 2 + ti - 24) * 4 + ((b & 7) >> 1);

  if (wave >= (int)NCONS) {
    __builtin_amdgcn_s_setprio(2);
    if (wave == (int)NCONS) loader<false>(p, L, lane_id(), wave, rg, cb);
    else loader<true>(p, L, lane_id(), wave, rg, cb);
    return;
  }

  {
    int lane = lane_id();
    if (wave == 0) ((volatile LAS u32*)(L + RING))[lane] = 0;
    wait_lgkm0();
    __builtin_amdgcn_s_barrier();
    if (wave == 0) STAMP(0);
    const int wr = wave >> 1, wc = wave & 1;
    u32 aaddr, aaddr1, baddr;
    {
      const int h = lane >> 5, c = lane & 31;
      aaddr = L_X + wr * 4096 + c * 64 + ((h ^ (c >> 2)) & 3) * 16;
      aaddr1 = aaddr ^ 32;
      baddr = (c & 3) * 4096 + h * 256 + (((8 * wc + (c >> 2)) ^ ((c & 3) << 2)) * 16);
    }
    const u32 saddr = L_SYNC + wave * 16;
    const __amdgpu_buffer_rsrc_t ro = mkrsrc(p.out, (u32)MDIM * NDIM * 4);
    const u32 so = (rg * 320 + wr * 64) * (NDIM * 4) + (cb * 256 + wc * 128) * 4;
    u32 touch = __builtin_amdgcn_raw_buffer_load_b32(ro, lane & 15 ? ~0u : so + lane * (NDIM * 4), 0, 0);
    float scale = __builtin_bit_cast(float, rfl(__builtin_bit_cast(u32, p.sx[0] * p.sw[0])));
    asm volatile("" : "+s"(scale) : "v"(touch));
    i32x16 acc[2][4];
#define MM(t, A, B) "v_mfma_i32_32x32x32_i8 v[" #t "*16:" #t "*16+15], v[" #A ":" #A "+3], v[" #B ":" #B "+3], v[" #t "*16:" #t "*16+15]\n" \
                    "v_mfma_i32_32x32x32_i8 v[" #t "*16+64:" #t "*16+79], v[" #A "+4:" #A "+7], v[" #B ":" #B "+3], v[" #t "*16+64:" #t "*16+79]\n"
#define RDA(A, adr) "ds_read_b128 v[" #A ":" #A "+3], v" #adr "\n" \
                    "ds_read_b128 v[" #A "+4:" #A "+7], v" #adr " offset:2048\n"
#define RDB(t, kh) "ds_read_b128 v[144+4*" #t ":147+4*" #t "], v161 offset:" #t "*1024+" #kh "*512\n"
#define WL(n) "s_waitcnt lgkmcnt(" #n ")\n"
    u32 sq, sav, ssp;
    asm volatile(
        "v_mov_b32 v128, 0\n v_mov_b32 v129, 0\n v_mov_b32 v130, 0\n v_mov_b32 v131, 0\n"
        "s_mov_b32 %[q], 0\n"
        "s_mov_b32 %[sp], 0\n"
        "s_setprio 1\n"
        "s_nop 1\n"
        "v_mfma_i32_32x32x32_i8 v[0:15], v[128:131], v[128:131], 0\n"
        "v_mfma_i32_32x32x32_i8 v[16:31], v[128:131], v[128:131], 0\n"
        "v_mfma_i32_32x32x32_i8 v[32:47], v[128:131], v[128:131], 0\n"
        "v_mfma_i32_32x32x32_i8 v[48:63], v[128:131], v[128:131], 0\n"
        "v_mfma_i32_32x32x32_i8 v[64:79], v[128:131], v[128:131], 0\n"
        "v_mfma_i32_32x32x32_i8 v[80:95], v[128:131], v[128:131], 0\n"
        "v_mfma_i32_32x32x32_i8 v[96:111], v[128:131], v[128:131], 0\n"
        "v_mfma_i32_32x32x32_i8 v[112:127], v[128:131], v[128:131], 0\n"
        "Lspin%=:\n"
        "ds_read_b64 v[164:165], v162\n"
        WL(0)
        "v_min_u32 v164, v164, v165\n"
        "s_nop 0\n"
        "v_readfirstlane_b32 %[av], v164\n"
        "s_cmp_gt_u32 %[av], %[q]\n"
        "s_cbranch_scc1 Lgo%=\n"
        "s_add_u32 %[sp], %[sp], 1\n"
        "s_cmp_gt_u32 %[sp], %[lim]\n"
        "s_cbranch_scc1 Lend%=\n"
        "s_sleep 1\n"
        "s_branch Lspin%=\n"
        "Lgo%=:\n"
        RDA(128, 160) RDB(0, 0) RDB(1, 0) RDB(2, 0)
        "Lstep%=:\n"
        RDB(3, 0) RDA(136, 166)
        "s_add_u32 %[q], %[q], 1\n"
        "v_mov_b32 v163, %[q]\n"
        WL(5) MM(0, 128, 144)
        RDB(0, 1) WL(5) MM(1, 128, 148)
        RDB(1, 1) WL(5) MM(2, 128, 152)
        RDB(2, 1) "ds_read_b64 v[164:165], v162\n" WL(6) MM(3, 128, 156)
        RDB(3, 1) "ds_write_b32 v162, v163 offset:8\n"
        "s_cmp_lg_u32 %[q], %[lastq]\n"
        "s_cbranch_scc1 Lkeep%=\n"
        "s_setprio 0\n"
        "Lkeep%=:\n"
        "s_and_b32 %[av], %[q], 3\n"
        "s_cselect_b32 %[av], 0x5000, %[wrap]\n"
        "v_add_u32 v161, 0x4000, v161\n"
        "v_add_u32 v160, %[av], v160\n"
        "v_add_u32 v166, %[av], v166\n"
        "v_and_b32 v161, 0xffff, v161\n"
        WL(5) MM(0, 136, 144)
        WL(2)
        "v_min_u32 v164, v164, v165\n"
        "s_nop 0\n"
        "v_readfirstlane_b32 %[av], v164\n"
        "s_cmp_gt_u32 %[av], %[q]\n"
        "s_cbranch_scc0 Lslow%=\n"
        RDA(128, 160) RDB(0, 0) MM(1, 136, 148)
        RDB(1, 0) MM(2, 136, 152)
        RDB(2, 0) WL(6) MM(3, 136, 156)
        "s_branch Lstep%=\n"
        "Lslow%=:\n"
        MM(1, 136, 148) MM(2, 136, 152) WL(0) MM(3, 136, 156)
        "s_cmp_lt_u32 %[q], 64\n"
        "s_cbranch_scc1 Lspin%=\n"
        "Lend%=:\n"
        "s_nop 15\n s_nop 7\n"
        : "={v[0:15]}"(acc[0][0]), "={v[16:31]}"(acc[0][1]), "={v[32:47]}"(acc[0][2]), "={v[48:63]}"(acc[0][3]),
          "={v[64:79]}"(acc[1][0]), "={v[80:95]}"(acc[1][1]), "={v[96:111]}"(acc[1][2]), "={v[112:127]}"(acc[1][3]),
          "+{v160}"(aaddr), "+{v166}"(aaddr1), "+{v161}"(baddr), [q] "=&s"(sq), [av] "=&s"(sav), [sp] "=&s"(ssp)
        : "{v162}"(saddr), [lim] "s"(SPIN_LIMIT), [wrap] "s"(0u - 3 * XUNIT), [lastq] "s"(DBG(7) ? 100u : NQ - NSLOT)
        : "memory", "scc", "v128", "v129", "v130", "v131", "v132", "v133", "v134", "v135", "v136", "v137", "v138", "v139", "v140",
          "v141", "v142", "v143", "v144", "v145", "v146", "v147", "v148", "v149", "v150", "v151", "v152", "v153", "v154", "v155",
          "v156", "v157", "v158", "v159", "v163", "v164", "v165" TRCCLOB);
    lane = lane_id();
    if (wave == 0) STAMP(2);
    if (rfl(ABORT[0]) != 0 || sq != NQ) scale = __builtin_nanf("");
    if (rg * 5 + wr >= 64) return;
    const u32 vo = so + (lane >> 5) * 4 * (NDIM * 4) + (lane & 31) * 16;
#pragma unroll
    for (int m = 0; m < 2; ++m)
#pragma unroll
      for (int r = 0; r < 16; ++r) {
        f32x4 v;
#pragma unroll
        for (int t = 0; t < 4; ++t) v[t] = __builtin_elementwise_maximum((float)acc[m][t][r] * scale, 0.f);
        __builtin_amdgcn_raw_buffer_store_b128(__builtin_bit_cast(u32x4, v), ro,
                                               vo + (m * 32 + (r & 3) + 8 * (r >> 2)) * (NDIM * 4), 0, 16);
      }
  }
}

}

extern "C" void kernel_launch(int rank, int nranks, ncclComm_t comm,
                              hipStream_t stream,
                              void* const* d_in, const int* in_sizes, int n_in,
                              void* d_out, int out_size,
                              const int* in_dtypes, int out_dtype,
                              void* const* d_out_all,
                              void* const* d_ws_all, size_t ws_bytes) {
  if (nranks != NR || n_in != 4 || (unsigned)rank >= (unsigned)NR || ws_bytes < WS_NEED ||
      in_sizes[0] != MDIM * KSH || in_sizes[1] != KSH * NDIM || out_size != MDIM * NDIM) {
    fprintf(stderr, "kernel_launch: unexpected problem shape\n");
    return;
  }
  Params p;
  p.src[0] = (const char*)d_in[0];
  p.src[1] = (const char*)d_in[1];
  p.sx = (const float*)d_in[2];
  p.sw = (const float*)d_in[3];
  p.out = (float*)d_out;
  p.ws0 = (char*)d_ws_all[rank];
  p.perm = 0;
  for (int d = 1, lo = 1, hi = 4; d < NR; ++d) {
    const bool same = ((0x4d >> rank) & 1) == ((0x4d >> ((rank + d) % NR)) & 1);
    p.perm |= (unsigned)d << (4 * (same ? lo++ : hi++));
  }
  for (int i = 0; i < NR; ++i) p.dst[i] = (char*)d_ws_all[(rank + ((p.perm >> (4 * i)) & 7)) % NR];
  fused_ag_gemm<<<NWG, NTHR, 0, stream>>>(p);
  HIP_CHECK(hipGetLastError());
}
```
